```python
import numpy as np

kernel_name = "gpu_gemm_a2a_m4096_k4096_n16384_bf16_relu"


ABSMAX_RTOL = 0.008

COMPUTE_DTYPE = "bfloat16"
EPILOGUE = "relu"
IN_QUANT = None
OUT_QUANT = None
IN_DTYPES = {"x": "bfloat16", "w_mat": "bfloat16"}
WORLD_SIZE = 8

def _to_bf16(x):
    u = np.ascontiguousarray(x, dtype=np.float32).view(np.uint32)
    return ((u + 0x7FFF + ((u >> 16) & 1)) & 0xFFFF0000).view(np.float32)

def setup_inputs():

    x = _to_bf16(np.random.randn(4096, 4096).astype(np.float32))
    w_mat = _to_bf16((np.random.randn(4096, 16384) / np.sqrt(4096)).astype(np.float32))
    return {"x": x, "w_mat": w_mat}

def _epilogue(y):
    return np.maximum(y, 0.0).astype(np.float32)

def reference(x, w_mat):
    y = (x @ w_mat).astype(np.float32)
    return _epilogue(y)
```

```cpp
#include <hip/hip_runtime.h>
#include <rccl/rccl.h>
#include <cstdio>

#define HIP_CHECK(expr) do { hipError_t _e = (expr); if (_e != hipSuccess) { \
    fprintf(stderr, "HIP error %s at %s:%d\n", hipGetErrorString(_e), __FILE__, __LINE__); \
    return; } } while (0)

namespace {

typedef unsigned short u16;
typedef unsigned long long u64;
typedef unsigned u32x4 __attribute__((ext_vector_type(4)));
typedef short s16x4 __attribute__((ext_vector_type(4)));
typedef short s16x8 __attribute__((ext_vector_type(8)));
typedef __bf16 bf16x8 __attribute__((ext_vector_type(8)));
typedef float f32x4 __attribute__((ext_vector_type(4)));
#define LAS __attribute__((address_space(3)))

constexpr int NR = 8;
constexpr int MS = 512, KD = 4096, ND = 16384, NS = 2048;
#define MF1 5
constexpr int MF2 = 8 - MF1;
constexpr int M1 = 32 * MF1, M2 = 32 * MF2;
constexpr int M1ROWS = 2 * M1, XROWS = 2 * M2;
constexpr int NKT = KD / 64;
#define NDMA 2
constexpr int NWAVES = 5 + NDMA;
constexpr int NTHREADS = NWAVES * 64;
constexpr int AP1 = M1 / 8, AP2 = M2 / 8;
constexpr int PPW1 = (AP1 + 16) / NDMA, PPW2 = (AP2 + 16) / NDMA;
constexpr int SLOT1 = (AP1 + 16) * 1024, SLOT2 = (AP2 + 16) * 1024;
constexpr int STAGE_OFF = 4 * SLOT2;
constexpr int CTRL_OFF = (4 * SLOT1 > STAGE_OFF + M1 * 256) ? 4 * SLOT1 : STAGE_OFF + M1 * 256;
constexpr int LDS_BYTES = CTRL_OFF + 16;
static_assert((AP1 + 16) % NDMA == 0 && (AP2 + 16) % NDMA == 0, "pieces");
static_assert(SLOT2 <= SLOT1 && LDS_BYTES <= 163840, "lds");
static_assert(3 * PPW1 <= 63, "vmcnt");
constexpr int XBLK = XROWS * KD * 2;
constexpr int XPC = XBLK / 256 / 1024;
static_assert(XPC * 256 * 1024 == XBLK, "x chunk");
constexpr int OPC = M1 / 4;

constexpr size_t WS_CALL = 0;
constexpr size_t WS_TMO = 4096;
constexpr size_t WS_XC = 8192;
constexpr size_t WS_OC = 16384;
constexpr size_t WS_STAMP = 65536;
constexpr size_t WS_G = 1u << 20;
constexpr size_t WS_NEED = WS_G + (size_t)2 * NR * XBLK;

constexpr u64 DEADLINE = 100000000ull;
#define XDT 64
#define ODT 57

struct Params {
  const u16* x; const u16* w;
  char* out[NR]; char* ws[NR];
  int rank;
};

#define WAIT_L0() asm volatile("s_waitcnt lgkmcnt(0)" ::: "memory")
#define BAR() do { asm volatile("" ::: "memory"); __builtin_amdgcn_s_barrier(); asm volatile("" ::: "memory"); } while (0)
template <int N> __device__ __forceinline__ void waitv() { asm volatile("s_waitcnt vmcnt(%0)" :: "i"(N) : "memory"); }

__device__ __forceinline__ void glds(const char* sbase, unsigned voff, unsigned ldsaddr) {
  asm volatile("s_mov_b32 m0, %2\n\ts_nop 0\n\tglobal_load_lds_dwordx4 %0, %1" :: "v"(voff), "s"(sbase), "s"(ldsaddr) : "memory");
}
__device__ __forceinline__ void glds_sys(const char* sbase, unsigned voff, unsigned ldsaddr) {
  asm volatile("s_mov_b32 m0, %2\n\ts_nop 0\n\tglobal_load_lds_dwordx4 %0, %1 sc0 sc1" :: "v"(voff), "s"(sbase), "s"(ldsaddr) : "memory");
}
__device__ __forceinline__ void store_sys(char* sbase, unsigned voff, u32x4 v) {
  asm volatile("global_store_dwordx4 %0, %1, %2 sc0 sc1\n\ts_nop 1" :: "v"(voff), "v"(v), "s"(sbase) : "memory");
}
__device__ __forceinline__ unsigned cvt_pk_bf16(float lo, float hi) {
  unsigned r; asm("v_cvt_pk_bf16_f32 %0, %1, %2" : "=v"(r) : "v"(lo), "v"(hi)); return r;
}
__device__ __forceinline__ float relu_nan(float v) { return v < 0.f ? 0.f : v; }
template <class T> __device__ __forceinline__ T* uni_ptr(T* p) {
  u64 v = (u64)p;
  unsigned lo = __builtin_amdgcn_readfirstlane((unsigned)v), hi = __builtin_amdgcn_readfirstlane((unsigned)(v >> 32));
  return (T*)(((u64)hi << 32) | lo);
}

__device__ __forceinline__ bool wait_ge(const unsigned* flag, unsigned want, unsigned* tmo, unsigned call) {
  const u64 t0 = __builtin_amdgcn_s_memrealtime();
  while ((unsigned)__builtin_amdgcn_readfirstlane(__hip_atomic_load(flag, __ATOMIC_RELAXED, __HIP_MEMORY_SCOPE_SYSTEM)) < want) {
    if ((unsigned)__builtin_amdgcn_readfirstlane(__hip_atomic_load(tmo, __ATOMIC_RELAXED, __HIP_MEMORY_SCOPE_AGENT)) == call ||
        __builtin_amdgcn_s_memrealtime() - t0 > DEADLINE) {
      __hip_atomic_store(tmo, call, __ATOMIC_RELAXED, __HIP_MEMORY_SCOPE_AGENT);
      return false;
    }
    __builtin_amdgcn_s_sleep(2);
  }
  return true;
}

template <int MF> struct Frag { bf16x8 a[MF]; bf16x8 b[2][2]; };

template <int MF>
__device__ __forceinline__ void read_frags(Frag<MF>& f, LAS char* lds, unsigned sb, int h, unsigned ra, const unsigned (&rb)[2][2]) {
  LAS char* pb = lds + sb + MF * 4096 + h * 8192;
#pragma unroll
  for (int b = 0; b < 2; ++b)
#pragma unroll
    for (int n = 0; n < 2; ++n) {
      s16x4 lo = __builtin_amdgcn_ds_read_tr16_b64_v4i16((LAS s16x4*)(pb + rb[b][n]));
      s16x4 hi = __builtin_amdgcn_ds_read_tr16_b64_v4i16((LAS s16x4*)(pb + rb[b][n] + 1024));
      f.b[b][n] = __builtin_bit_cast(bf16x8, (s16x8)__builtin_shufflevector(lo, hi, 0, 1, 2, 3, 4, 5, 6, 7));
    }
  LAS char* pa = lds + sb + (ra ^ (unsigned)(h * 64));
#pragma unroll
  for (int mi = 0; mi < MF; ++mi) f.a[mi] = *(LAS bf16x8*)(pa + mi * 2048);
}
template <int MF>
__device__ __forceinline__ void mma(f32x4 (&acc)[MF][2][2], const Frag<MF>& f) {
#pragma unroll
  for (int mi = 0; mi < MF; ++mi)
#pragma unroll
    for (int b = 0; b < 2; ++b)
#pragma unroll
      for (int n = 0; n < 2; ++n)
        acc[mi][b][n] = __builtin_amdgcn_mfma_f32_16x16x32_bf16(f.b[b][n], f.a[mi], acc[mi][b][n], 0, 0, 0);
}
template <int NM, int NRD> __device__ __forceinline__ void interleave() {
#pragma unroll
  for (int i = 0; i < (NRD < NM ? NRD : NM); ++i) {
    __builtin_amdgcn_sched_group_barrier(0x8, 1, 0);
    __builtin_amdgcn_sched_group_barrier(0x100, 1, 0);
  }
  if (NRD > NM) __builtin_amdgcn_sched_group_barrier(0x100, NRD - NM, 0);
  if (NM > NRD) __builtin_amdgcn_sched_group_barrier(0x8, NM - NRD, 0);
}
__device__ __forceinline__ u32x4 pack8(const f32x4& v0, const f32x4& v1, bool odd) {
  unsigned a0 = cvt_pk_bf16(relu_nan(v0[0]), relu_nan(v0[1])), a1 = cvt_pk_bf16(relu_nan(v0[2]), relu_nan(v0[3]));
  unsigned b0 = cvt_pk_bf16(relu_nan(v1[0]), relu_nan(v1[1])), b1 = cvt_pk_bf16(relu_nan(v1[2]), relu_nan(v1[3]));
  u32x4 o;
  o[0] = odd ? b0 : a0; o[1] = odd ? b1 : a1; o[2] = odd ? a0 : b0; o[3] = odd ? a1 : b1;
  return o;
}

#define STAMP(i) do {} while (0)
#define STAMPC(i) do {} while (0)

__global__ void __launch_bounds__(NTHREADS, (NWAVES + 3) / 4) fused_gemm_a2a(const Params P) {
  extern __shared__ __attribute__((aligned(16))) char smem_[];
  LAS char* lds = (LAS char*)smem_;
  const int tid = threadIdx.x, lane = tid & 63, wid = __builtin_amdgcn_readfirstlane(tid >> 6);
  const int bid = blockIdx.x, xcd = bid & 7, idx = bid >> 3, rank = P.rank;
  const int mh = idx & 1, nt1 = (idx >> 1) * 8 + xcd, dst1 = nt1 >> 4;
  const int mt2 = (xcd & 1) * 8 + (idx & 7), nl2 = (xcd >> 1) * 4 + (idx >> 3), src2 = mt2 >> 1, half2 = mt2 & 1;
  char* myws = P.ws[rank];
  unsigned* tmo = (unsigned*)(myws + WS_TMO);

  if (wid < 4) {
    const int wr = wid >> 1, wc = wid & 1, fr = lane & 15, fq = lane >> 4, q = fr >> 2, p = lane & 3;
    const unsigned rb00 = (8 * fq + q) * 256 + ((((2 * wc) ^ q) << 2) | (p ^ (fq & 1))) * 16 + (p & 1) * 8;
    const unsigned rb[2][2] = {{rb00, rb00 ^ 8u}, {rb00 ^ 64u, rb00 ^ 72u}};
    const unsigned rsw = (unsigned)((fq ^ (fr >> 1)) * 16);
    const unsigned ra1 = (wr * (M1 / 2) + fr) * 128 + rsw, ra2 = (wr * (M2 / 2) + fr) * 128 + rsw;
    const bool odd = fq & 1;
    if (wid == 0) { STAMP(0); STAMPC(8); }
    Frag<MF2> g0, g1;
    {
      f32x4 acc[MF1][2][2];
#pragma unroll
      for (int i = 0; i < MF1; ++i)
#pragma unroll
        for (int b = 0; b < 2; ++b)
#pragma unroll
          for (int n = 0; n < 2; ++n) acc[i][b][n] = (f32x4){0.f, 0.f, 0.f, 0.f};
      Frag<MF1> f0, f1;
      BAR();
      if (wid == 0) STAMP(1);
      read_frags<MF1>(f0, lds, 0, 0, ra1, rb);
      for (int i = 0; i < NKT - 1; ++i) {
        const unsigned sb = (i & 3) * SLOT1, sn = ((i + 1) & 3) * SLOT1;
        read_frags<MF1>(f1, lds, sb, 1, ra1, rb);
        mma<MF1>(acc, f0);
        interleave<4 * MF1, 8 + MF1>();
        __builtin_amdgcn_sched_barrier(0);
        WAIT_L0(); BAR();
        __builtin_amdgcn_sched_barrier(0);
        read_frags<MF1>(f0, lds, sn, 0, ra1, rb);
        mma<MF1>(acc, f1);
        interleave<4 * MF1, 8 + MF1>();
        __builtin_amdgcn_sched_barrier(0);
      }
      read_frags<MF1>(f1, lds, 3 * SLOT1, 1, ra1, rb);
      mma<MF1>(acc, f0);
      interleave<4 * MF1, 8 + MF1>();
      __builtin_amdgcn_sched_barrier(0);
      WAIT_L0(); BAR();
      __builtin_amdgcn_sched_barrier(0);
      read_frags<MF2>(g0, lds, 0, 0, ra2, rb);
      mma<MF1>(acc, f1);
      interleave<4 * MF1, 8 + MF2>();
      __builtin_amdgcn_sched_barrier(0);
      if (wid == 0) STAMP(2);
#pragma unroll
      for (int mi = 0; mi < MF1; ++mi)
#pragma unroll
        for (int b = 0; b < 2; ++b) {
          const int row = wr * (M1 / 2) + mi * 16 + fr;
          *(LAS u32x4*)(lds + STAGE_OFF + row * 256 + (((wc * 8 + b * 4 + fq) ^ (fr & 7)) * 16)) = pack8(acc[mi][b][0], acc[mi][b][1], odd);
        }
    }
    f32x4 acc[MF2][2][2];
#pragma unroll
    for (int i = 0; i < MF2; ++i)
#pragma unroll
      for (int b = 0; b < 2; ++b)
#pragma unroll
        for (int n = 0; n < 2; ++n) acc[i][b][n] = (f32x4){0.f, 0.f, 0.f, 0.f};
    __builtin_amdgcn_sched_barrier(0);
    for (int i = 0; i < NKT - 1; ++i) {
      const unsigned sb = (i & 3) * SLOT2, sn = ((i + 1) & 3) * SLOT2;
      read_frags<MF2>(g1, lds, sb, 1, ra2, rb);
      mma<MF2>(acc, g0);
      interleave<4 * MF2, 8 + MF2>();
      __builtin_amdgcn_sched_barrier(0);
      WAIT_L0(); BAR();
      __builtin_amdgcn_sched_barrier(0);
      read_frags<MF2>(g0, lds, sn, 0, ra2, rb);
      mma<MF2>(acc, g1);
      interleave<4 * MF2, 8 + MF2>();
      __builtin_amdgcn_sched_barrier(0);
    }
    read_frags<MF2>(g1, lds, 3 * SLOT2, 1, ra2, rb);
    mma<MF2>(acc, g0);
    interleave<4 * MF2, 8 + MF2>();
    __builtin_amdgcn_sched_barrier(0);
    WAIT_L0(); BAR();
    __builtin_amdgcn_sched_barrier(0);
    const unsigned bad = *(LAS unsigned*)(lds + CTRL_OFF + 4);
    mma<MF2>(acc, g1);
    if (wid == 0) STAMP(3);
    char* ob = P.out[rank] + ((size_t)(src2 * MS + M1ROWS + half2 * M2 + wr * (M2 / 2) + fr) * NS + nl2 * 128 + wc * 64 + fq * 8) * 2;
#pragma unroll
    for (int mi = 0; mi < MF2; ++mi)
#pragma unroll
      for (int b = 0; b < 2; ++b) {
        u32x4 o = pack8(acc[mi][b][0], acc[mi][b][1], odd);
        if (bad) o = (u32x4){0x7fc07fc0u, 0x7fc07fc0u, 0x7fc07fc0u, 0x7fc07fc0u};
        *(u32x4*)(ob + (size_t)mi * 16 * NS * 2 + b * 64) = o;
      }
  } else if (wid < 4 + NDMA) {
    const int dw = wid - 4;
    unsigned va[2], vb[2];
#pragma unroll
    for (int c = 0; c < 2; ++c) {
      va[c] = (unsigned)((lane >> 3) * (KD * 2) + (((lane & 7) ^ ((4 * c + (lane >> 4)) & 7)) * 16));
      vb[c] = (unsigned)((lane >> 4) * (ND * 2) + (((lane & 15) ^ (((lane >> 4) << 2) | c)) * 16));
    }
    const unsigned vA = (NDMA == 1) ? 0u : ((dw & 1) ? va[1] : va[0]);
    const unsigned vB4 = ((dw >> 1) & 1) ? vb[1] : vb[0];
    const unsigned ldsb = (unsigned)(size_t)lds;
    const char* a1 = (const char*)P.x + (size_t)(mh * M1 + dw * 8) * KD * 2;
    const char* w1 = (const char*)P.w + (size_t)nt1 * 256 + (size_t)(dw * 4) * ND * 2;
    const char* w2 = (const char*)P.w + (size_t)(rank * 16 + nl2) * 256 + (size_t)(dw * 4) * ND * 2;
    const char* a2 = nullptr;
    auto issue1 = [&](int kt) {
      const unsigned sb = ldsb + (kt & 3) * SLOT1 + dw * 1024;
      const char* wp = w1 + (size_t)kt * 64 * ND * 2;
      const char* ap = a1 + kt * 128;
#pragma unroll
      for (int i = 0; i < 16 / NDMA; ++i)
        glds(wp + (size_t)i * NDMA * 4 * ND * 2, NDMA == 4 ? vB4 : vb[i & 1], sb + AP1 * 1024 + i * NDMA * 1024);
#pragma unroll
      for (int i = 0; i < AP1 / NDMA; ++i) glds(ap + (size_t)i * NDMA * 8 * KD * 2, vA, sb + i * NDMA * 1024);
    };
    auto issue2 = [&](int kt) {
      const unsigned sb = ldsb + (kt & 3) * SLOT2 + dw * 1024;
      const char* wp = w2 + (size_t)kt * 64 * ND * 2;
      const char* ap = a2 + kt * 128;
#pragma unroll
      for (int i = 0; i < 16 / NDMA; ++i)
        glds(wp + (size_t)i * NDMA * 4 * ND * 2, NDMA == 4 ? vB4 : vb[i & 1], sb + AP2 * 1024 + i * NDMA * 1024);
#pragma unroll
      for (int i = 0; i < AP2 / NDMA; ++i) glds_sys(ap + (size_t)i * NDMA * 8 * KD * 2, vA, sb + i * NDMA * 1024);
    };
    issue1(0); issue1(1); issue1(2);
    waitv<2 * PPW1>();
    BAR();
    issue1(3);
    for (int i = 0; i < NKT - 4; ++i) {
      waitv<2 * PPW1>(); BAR(); issue1(i + 4);
    }
    waitv<2 * PPW1>(); BAR();
    {
      const unsigned call = (unsigned)__builtin_amdgcn_readfirstlane(*(LAS unsigned*)(lds + CTRL_OFF));
      a2 = (src2 == rank) ? (const char*)P.x + (size_t)(M1ROWS + half2 * M2 + dw * 8) * KD * 2
                          : myws + WS_G + ((size_t)((call & 1) * NR + src2) * XROWS + half2 * M2 + dw * 8) * KD * 2;
    }
    issue2(0);
    waitv<PPW1 + PPW2>(); BAR(); issue2(1);
    waitv<2 * PPW2>(); BAR(); issue2(2);
    waitv<2 * PPW2>(); BAR(); issue2(3);
    for (int i = 0; i < NKT - 4; ++i) {
      waitv<2 * PPW2>(); BAR(); issue2(i + 4);
    }
    waitv<2 * PPW2>(); BAR();
    waitv<PPW2>(); BAR();
    waitv<0>(); BAR();
    BAR();
  } else {
    unsigned* callp = (unsigned*)(myws + WS_CALL) + bid;
    const unsigned call = (unsigned)__builtin_amdgcn_readfirstlane(__hip_atomic_load(callp, __ATOMIC_RELAXED, __HIP_MEMORY_SCOPE_AGENT)) + 1u;
    u32x4 xv[XPC];
    {
      const char* xs = (const char*)P.x + (size_t)M1ROWS * KD * 2 + (size_t)bid * (XPC * 1024) + lane * 16;
#pragma unroll
      for (int i = 0; i < XPC; ++i) xv[i] = *(const u32x4*)(xs + i * 1024);
#pragma unroll
      for (int i = 0; i < XPC; ++i) asm volatile("" : "+v"(xv[i]));
    }
    const size_t goff = WS_G + (size_t)((call & 1) * NR + rank) * XBLK + (size_t)bid * (XPC * 1024);
    const size_t xcoff = WS_XC + (size_t)(rank * 2 + (bid >> 7)) * 128;
    const unsigned vl = lane * 16;
    int nbar = 0;
    u64 tnext = __builtin_amdgcn_s_memrealtime();
    int barlim = NKT - 3;
    auto pace = [&]() {
      while (__builtin_amdgcn_s_memrealtime() < tnext) {
        if (nbar < barlim) { BAR(); ++nbar; } else __builtin_amdgcn_s_sleep(1);
      }
    };
    const int rot = bid % 7;
    for (int j = 0; j < NR - 1; ++j) {
      int pj = j + rot; if (pj >= NR - 1) pj -= NR - 1;
      const int peer = (rank + 1 + pj) & (NR - 1);
      char* g = uni_ptr(P.ws[peer]) + goff;
#pragma unroll
      for (int i = 0; i < XPC; ++i) {
        pace();
        store_sys(g + i * 1024, vl, xv[i]);
        tnext += XDT;
        if (i == 2 && j > 0) {
          waitv<3>();
          int pp = j - 1 + rot; if (pp >= NR - 1) pp -= NR - 1;
          const int prev = (rank + 1 + pp) & (NR - 1);
          if (lane == 0) __hip_atomic_fetch_add((unsigned*)(P.ws[prev] + xcoff), 1u, __ATOMIC_RELAXED, __HIP_MEMORY_SCOPE_SYSTEM);
        }
      }
    }
    tnext += 60;
    pace();
    waitv<0>();
    {
      int pp = NR - 2 + rot; if (pp >= NR - 1) pp -= NR - 1;
      const int prev = (rank + 1 + pp) & (NR - 1);
      if (lane == 0) __hip_atomic_fetch_add((unsigned*)(P.ws[prev] + xcoff), 1u, __ATOMIC_RELAXED, __HIP_MEMORY_SCOPE_SYSTEM);
    }
    STAMP(6);
    while (nbar < NKT - 3) { BAR(); ++nbar; }
    {
      bool ok = true;
      if (src2 != rank) ok = wait_ge((const unsigned*)(myws + WS_XC + (src2 * 2 + half2) * 128), call * 128u, tmo, call);
      if (lane == 0) { *(LAS unsigned*)(lds + CTRL_OFF) = call; *(LAS unsigned*)(lds + CTRL_OFF + 4) = ok ? 0u : 1u; }
      WAIT_L0();
    }
    STAMP(5);
    barlim = 2 * NKT + 1;
    while (nbar < NKT + 2) { BAR(); ++nbar; }
    {
      char* o = uni_ptr(P.out[dst1]) + ((size_t)(rank * MS + mh * M1) * NS + (nt1 & 15) * 128) * 2;
      const unsigned vo = (unsigned)((lane >> 4) * (NS * 2) + (lane & 15) * 16);
      const unsigned sr0 = STAGE_OFF + (lane >> 4) * 256 + (((lane & 15) ^ (lane >> 4)) * 16);
      const bool remote = dst1 != rank;
      u64 now = __builtin_amdgcn_s_memrealtime();
      if (tnext < now) tnext = now;
      for (int i = 0; i < OPC; ++i) {
        u32x4 v = *(LAS u32x4*)(lds + ((sr0 + i * 1024) ^ ((i & 1) * 64u)));
        pace();
        store_sys(o + (size_t)i * 4 * NS * 2, vo, v);
        tnext += remote ? ODT : 20;
      }
      while (nbar < 2 * NKT + 1) { BAR(); ++nbar; }
      waitv<0>();
      if (remote && lane == 0)
        __hip_atomic_fetch_add((unsigned*)(P.ws[dst1] + WS_OC + rank * 128), 1u, __ATOMIC_RELAXED, __HIP_MEMORY_SCOPE_SYSTEM);
    }
    STAMP(7);
    if (dst1 == rank && (nt1 & 15) == 0 && mh == 0) {
      bool ok = true;
      for (int s = 0; s < NR; ++s)
        if (s != rank) ok = wait_ge((const unsigned*)(myws + WS_OC + s * 128), call * 32u, tmo, call) && ok;
      __builtin_amdgcn_fence(__ATOMIC_ACQUIRE, "");
      if (!ok && lane == 0) {
        __hip_atomic_store((unsigned*)(P.out[rank] + (size_t)(rank * MS) * NS * 2), 0x7fc07fc0u, __ATOMIC_RELAXED, __HIP_MEMORY_SCOPE_SYSTEM);
      }
      STAMP(10);
    }
    if (lane == 0) __hip_atomic_store(callp, call, __ATOMIC_RELAXED, __HIP_MEMORY_SCOPE_AGENT);
  }
}

}

extern "C" void kernel_launch(int rank, int nranks, ncclComm_t comm,
                              hipStream_t stream,
                              void* const* d_in, const int* in_sizes, int n_in,
                              void* d_out, int out_size,
                              const int* in_dtypes, int out_dtype,
                              void* const* d_out_all,
                              void* const* d_ws_all, size_t ws_bytes) {
  if (nranks != NR || (unsigned)rank >= (unsigned)NR || ws_bytes < WS_NEED || n_in < 2) {
    fprintf(stderr, "unsupported configuration\n");
    return;
  }
  static bool init[NR] = {};
  if (!init[rank]) {
    HIP_CHECK(hipFuncSetAttribute((const void*)fused_gemm_a2a, hipFuncAttributeMaxDynamicSharedMemorySize, LDS_BYTES));
    init[rank] = true;
  }
  Params P;
  P.x = (const u16*)d_in[0];
  P.w = (const u16*)d_in[1];
  for (int p = 0; p < NR; ++p) { P.out[p] = (char*)d_out_all[p]; P.ws[p] = (char*)d_ws_all[p]; }
  P.rank = rank;
  hipLaunchKernelGGL(fused_gemm_a2a, dim3(256), dim3(NTHREADS), LDS_BYTES, stream, P);
  HIP_CHECK(hipGetLastError());
}
```

```cpp
#include <hip/hip_runtime.h>
#include <rccl/rccl.h>
#include <cstdio>

#define HIP_CHECK(expr) do { hipError_t _e = (expr); if (_e != hipSuccess) { \
    fprintf(stderr, "HIP error %s at %s:%d\n", hipGetErrorString(_e), __FILE__, __LINE__); \
    return; } } while (0)

namespace {

typedef unsigned short u16;
typedef unsigned long long u64;
typedef unsigned u32x4 __attribute__((ext_vector_type(4)));
typedef short s16x4 __attribute__((ext_vector_type(4)));
typedef short s16x8 __attribute__((ext_vector_type(8)));
typedef __bf16 bf16x8 __attribute__((ext_vector_type(8)));
typedef float f32x4 __attribute__((ext_vector_type(4)));
#define LAS __attribute__((address_space(3)))

constexpr int NR = 8;
constexpr int MS = 512, KD = 4096, ND = 16384, NS = 2048;
constexpr int MF1 = 5, MF2 = 4;
constexpr int M1 = 32 * MF1, M2 = 32 * MF2;
constexpr int M1ROWS = 2 * M1, XROWS = MS - M1ROWS;
constexpr int NKT = KD / 64;
constexpr int NDMA = 2;
constexpr int NWAVES = 7;
constexpr int NTHREADS = NWAVES * 64;
constexpr int AP1 = M1 / 8, AP2 = M2 / 8;
constexpr int PPW1 = (AP1 + 16) / NDMA, PPW2 = (AP2 + 16) / NDMA;
constexpr int A_OFF = 65536, ASL1 = M1 * 128, ASL2 = M2 * 128;
constexpr int LDS_BYTES = A_OFF + 4 * ASL1;
static_assert(3 * PPW1 <= 63 && ASL2 <= ASL1 && LDS_BYTES <= 163840, "rings");
constexpr int XBLK = XROWS * KD * 2;
constexpr int OPC = M1 / 4;
constexpr int TILE1 = OPC * 1024;
static_assert(XROWS == 6 * 32 && NR * XROWS == 12 * M2, "groups");

constexpr size_t WS_CALL = 0;
constexpr size_t WS_TMO = 4096;
constexpr size_t WS_XC = 8192;
constexpr size_t WS_OF = 16384;
constexpr size_t WS_STAMP = 65536;
constexpr size_t WS_G = 1u << 20;
constexpr size_t WS_S = WS_G + (size_t)2 * NR * XBLK;
constexpr size_t WS_NEED = WS_S + (size_t)2 * 256 * TILE1;

constexpr u64 DEADLINE = 100000000ull;
#define XTHR 2
#define PDEP 1
#define WPFD 1
#define FLAGK 1

struct Params {
  const u16* x; const u16* w;
  char* out; char* ws[NR];
  int rank;
};

#define BAR() do { asm volatile("" ::: "memory"); __builtin_amdgcn_s_barrier(); asm volatile("" ::: "memory"); } while (0)
#define SCHED() __builtin_amdgcn_sched_barrier(0)
#define WAIT_LGKM(n) __builtin_amdgcn_s_waitcnt(0xC07F | ((n) << 8))
#define MFMA_DRAIN() asm volatile("s_nop 7\n\ts_nop 7\n\ts_nop 3" ::: "memory")
template <int N> __device__ __forceinline__ void waitv() { asm volatile("s_waitcnt vmcnt(%0)" :: "i"(N) : "memory"); }

template <int OFF, bool SYS> __device__ __forceinline__ void glds(const char* sbase, unsigned voff, unsigned ldsbase) {
#define A2FL " sc0 sc1"
  if (SYS) asm volatile("s_add_u32 m0, %2, %3\n\ts_nop 0\n\tglobal_load_lds_dwordx4 %0, %1" A2FL :: "v"(voff), "s"(sbase), "s"(ldsbase), "i"(OFF) : "memory", "scc");
  else asm volatile("s_add_u32 m0, %2, %3\n\ts_nop 0\n\tglobal_load_lds_dwordx4 %0, %1" :: "v"(voff), "s"(sbase), "s"(ldsbase), "i"(OFF) : "memory", "scc");
}
template <int I, int N, bool SYS> struct Pieces {
  static __device__ __forceinline__ void go(const char* sbase, const unsigned* voff, unsigned ldsbase) {
    glds<I * 1024 * NDMA, SYS>(sbase, voff[I], ldsbase);
    Pieces<I + 1, N, SYS>::go(sbase, voff, ldsbase);
  }
};
template <int N, bool SYS> struct Pieces<N, N, SYS> { static __device__ __forceinline__ void go(const char*, const unsigned*, unsigned) {} };

template <int OFF = 0> __device__ __forceinline__ void store_sys(char* sbase, unsigned voff, u32x4 v) {
  asm volatile("global_store_dwordx4 %0, %1, %2 offset:%3 sc0 sc1\n\ts_nop 1" :: "v"(voff), "v"(v), "s"(sbase), "i"(OFF) : "memory");
}
template <int OFF = 0> __device__ __forceinline__ void store_loc(char* sbase, unsigned voff, u32x4 v) {
  asm volatile("global_store_dwordx4 %0, %1, %2 offset:%3\n\ts_nop 1" :: "v"(voff), "v"(v), "s"(sbase), "i"(OFF) : "memory");
}
__device__ __forceinline__ unsigned cvt_pk_bf16(float lo, float hi) {
  unsigned r; asm("v_cvt_pk_bf16_f32 %0, %1, %2" : "=v"(r) : "v"(lo), "v"(hi)); return r;
}
__device__ __forceinline__ float relu_nan(float v) { return __builtin_elementwise_maximum(v, 0.f); }
__device__ __forceinline__ unsigned rfl(unsigned v) { return (unsigned)__builtin_amdgcn_readfirstlane(v); }
__device__ __forceinline__ unsigned vm_count() {
  unsigned r; asm volatile("s_getreg_b32 %0, hwreg(HW_REG_IB_STS)" : "=s"(r) :: "memory");
  return (r & 15u) | ((r >> 18) & 48u);
}
__device__ __forceinline__ unsigned sload(const char* p) {
  unsigned r; asm volatile("s_load_dword %0, %1, 0x0\n\ts_waitcnt lgkmcnt(0)" : "=s"(r) : "s"(p) : "memory"); return r;
}

__device__ __forceinline__ bool wait_ge(const unsigned* flag, unsigned want, unsigned* tmo, unsigned call) {
  const u64 t0 = __builtin_amdgcn_s_memrealtime();
  while (rfl(__hip_atomic_load(flag, __ATOMIC_RELAXED, __HIP_MEMORY_SCOPE_SYSTEM)) < want) {
    if (rfl(__hip_atomic_load(tmo, __ATOMIC_RELAXED, __HIP_MEMORY_SCOPE_AGENT)) == call ||
        __builtin_amdgcn_s_memrealtime() - t0 > DEADLINE) {
      __hip_atomic_store(tmo, call, __ATOMIC_RELAXED, __HIP_MEMORY_SCOPE_AGENT);
      waitv<0>();
      return false;
    }
    __builtin_amdgcn_s_sleep(2);
  }
  return true;
}

template <int MF> struct Frag { bf16x8 a[MF]; s16x4 lo[4], hi[4]; };

template <int MF>
__device__ __forceinline__ void read_one(Frag<MF>& f, int i, LAS char* lds, int s, int h, const unsigned (&ra)[2], const unsigned (&rb)[4]) {
  if (i >= 2 && i < 2 + MF) {
    f.a[i - 2] = *(LAS bf16x8*)(lds + ra[h] + s * (MF * 4096) + (i - 2) * 2048);
  } else {
    const int k = i < 2 ? i : i - MF, bn = k >> 1;
    s16x4 v = __builtin_amdgcn_ds_read_tr16_b64_v4i16((LAS s16x4*)(lds + rb[bn] + s * 16384 + h * 8192 + (k & 1) * 1024));
    if (k & 1) f.hi[bn] = v; else f.lo[bn] = v;
  }
}
template <int MF, int MG, bool RD>
__device__ __forceinline__ void half_step(f32x4 (&acc)[MF][4], const Frag<MF>& f, Frag<MG>& g, LAS char* lds, int s, int h,
                                          const unsigned (&ra)[2], const unsigned (&rb)[4]) {
#pragma unroll
  for (int i = 0; i < 4 * MF; ++i) {
    const int bn = i / MF, mi = i % MF;
    const s16x8 bv = __builtin_shufflevector(f.lo[bn], f.hi[bn], 0, 1, 2, 3, 4, 5, 6, 7);
    asm volatile("v_mfma_f32_16x16x32_bf16 %0, %1, %2, %0" : "+v"(acc[mi][bn]) : "v"(bv), "v"(f.a[mi]));
    if (RD && i < 8 + MG) read_one<MG>(g, i, lds, s, h, ra, rb);
    SCHED();
  }
}
template <int MF, int MG, int S, bool NEXT, bool DRAINV = false>
__device__ __forceinline__ void ktile(f32x4 (&acc)[MF][4], Frag<MF>& f0, Frag<MF>& f1, Frag<MG>& nx, LAS char* lds,
                                      const unsigned (&ra)[2], const unsigned (&rn)[2], const unsigned (&rb)[4]) {
  WAIT_LGKM(0);
  half_step<MF, MF, true>(acc, f0, f1, lds, S, 1, ra, rb);
  WAIT_LGKM(0);
  if (DRAINV) waitv<0>();
  BAR();
  SCHED();
  half_step<MF, MG, NEXT>(acc, f1, nx, lds, (S + 1) & 3, 0, rn, rb);
}
__device__ __forceinline__ u32x4 pack8(const f32x4& v0, const f32x4& v1, bool odd) {
  unsigned a0 = cvt_pk_bf16(relu_nan(v0[0]), relu_nan(v0[1])), a1 = cvt_pk_bf16(relu_nan(v0[2]), relu_nan(v0[3]));
  unsigned b0 = cvt_pk_bf16(relu_nan(v1[0]), relu_nan(v1[1])), b1 = cvt_pk_bf16(relu_nan(v1[2]), relu_nan(v1[3]));
  u32x4 o;
  o[0] = odd ? b0 : a0; o[1] = odd ? b1 : a1; o[2] = odd ? a0 : b0; o[3] = odd ? a1 : b1;
  return o;
}

#define STAMP(i) do {} while (0)
#define STAMPC(i) do {} while (0)

__global__ void __launch_bounds__(NTHREADS, 2) fused_gemm_a2a(const Params P) {
  extern __shared__ __attribute__((aligned(16))) char smem_[];
  LAS char* lds = (LAS char*)smem_;
  const int tid = threadIdx.x, lane = tid & 63, wid = __builtin_amdgcn_readfirstlane(tid >> 6);
  const int bid = blockIdx.x, xcd = bid & 7, idx = bid >> 3, rank = P.rank;
#define NTMAP 0
  const int mh = idx & 1, dst1 = idx >> 2, t32 = bid & 31;
  auto colt = [&](int owner) { const int hb = (t32 >> 4) & 1; return NTMAP == 0 ? hb * 8 + xcd : NTMAP == 1 ? xcd * 2 + hb : hb * 8 + ((xcd + owner * 2 + hb) & 7); };
  const int nt1 = dst1 * 16 + colt(dst1);
  const int rel = (dst1 - rank) & (NR - 1);
  const int ci = (rel - 2) * 4 + (idx & 3), m6 = ci % 6, n4 = ci / 6;
#define P2MAP 0
  const int mt2 = P2MAP == 0 ? (xcd & 1) * 6 + m6 : P2MAP == 1 ? ci % 12 : (xcd & 3) * 3 + ci % 3;
  const int nl2 = P2MAP == 0 ? (xcd >> 1) * 4 + n4 : P2MAP == 1 ? xcd * 2 + ci / 12 : (xcd >> 2) * 8 + ci / 3;
  char* myws = P.ws[rank];
  unsigned* tmo = (unsigned*)(myws + WS_TMO);
  const char* callp = myws + WS_CALL + bid * 4;
  if (wid == 0) { STAMP(0); STAMPC(8); }

  if (rel == 0) {
    static_assert(XTHR < NR - 1, "xthr");
    char* sigp;
    {
      const int peer = (rank + 1 + lane) & (NR - 1);
      char* pw = P.ws[0];
#pragma unroll
      for (int r = 1; r < NR; ++r) pw = (peer == r) ? P.ws[r] : pw;
      sigp = pw + WS_XC + rank * 128;
    }
    if (wid < 6) {
      const int row = t32 * 6 + wid;
      const unsigned vl = lane * 16;
      u32x4 xv[8];
      const char* xs = (const char*)P.x + (size_t)(M1ROWS + row) * KD * 2 + vl;
#pragma unroll
      for (int i = 0; i < 8; ++i) xv[i] = *(const u32x4*)(xs + i * 1024);
      const unsigned call = sload(callp) + 1u;
      const size_t goff = WS_G + (size_t)((call & 1) * NR + rank) * XBLK + (size_t)row * KD * 2;
      char* g[NR - 1];
#pragma unroll
      for (int j = 0; j < NR - 1; ++j) { int pj = j + wid; if (pj >= NR - 1) pj -= NR - 1; g[j] = P.ws[(rank + 1 + pj) & (NR - 1)] + goff; }
#pragma unroll
      for (int i = 0; i < 8; ++i) asm volatile("" : "+v"(xv[i]));
#pragma unroll
      for (int i = 0; i < 8; ++i) {
        unsigned vo = vl + i * 1024;
        asm volatile("" : "+v"(vo));
#pragma unroll
        for (int j = 0; j < NR - 1; ++j) { store_sys(g[j], vo, xv[i]); waitv<XTHR>(); }
        if (i == 4 && lane < NR - 1) __hip_atomic_fetch_add((unsigned*)sigp, 1u, __ATOMIC_RELAXED, __HIP_MEMORY_SCOPE_SYSTEM);
      }
      waitv<0>();
    }
    BAR();
    if (wid == 6) {
      if (lane < NR - 1) __hip_atomic_fetch_add((unsigned*)(sigp + NR * 128), 1u, __ATOMIC_RELAXED, __HIP_MEMORY_SCOPE_SYSTEM);
      STAMP(6);
    }
  }

  if (wid < 4) {
    const int wr = wid >> 1, wc = wid & 1, fr = lane & 15, fq = lane >> 4, q = fr >> 2, p = lane & 3;
    const unsigned rb00 = (8 * fq + q) * 256 + ((((2 * wc) ^ q) << 2) | (p ^ (fq & 1))) * 16 + (p & 1) * 8;
    const unsigned rb[4] = {rb00, rb00 ^ 8u, rb00 ^ 64u, rb00 ^ 72u};
    const unsigned rsw = (unsigned)((fq ^ (fr >> 1)) * 16);
    const unsigned ra1b = A_OFF + (wr * (M1 / 2) + fr) * 128 + rsw, ra2b = A_OFF + (wr * (M2 / 2) + fr) * 128 + rsw;
    const unsigned ra1[2] = {ra1b, ra1b ^ 64u}, ra2[2] = {ra2b, ra2b ^ 64u};
    const bool odd = fq & 1;
    const unsigned call = sload(callp) + 1u;
    const unsigned vrow = (unsigned)(fr * (NS * 2) + fq * 16);
    Frag<MF2> g0, g1;
    {
      f32x4 acc[MF1][4];
#pragma unroll
      for (int i = 0; i < MF1; ++i)
#pragma unroll
        for (int b = 0; b < 4; ++b) { acc[i][b] = (f32x4){0.f, 0.f, 0.f, 0.f}; asm volatile("" : "+v"(acc[i][b])); }
      Frag<MF1> f0, f1;
      BAR();
      if (wid == 0) STAMP(1);
      WAIT_LGKM(0);
#pragma unroll
      for (int i = 0; i < 8 + MF1; ++i) read_one<MF1>(f0, i, lds, 0, 0, ra1, rb);
      SCHED();
      for (int j = 0; j < NKT - 4; j += 4) {
        ktile<MF1, MF1, 0, true>(acc, f0, f1, f0, lds, ra1, ra1, rb);
        ktile<MF1, MF1, 1, true>(acc, f0, f1, f0, lds, ra1, ra1, rb);
        ktile<MF1, MF1, 2, true>(acc, f0, f1, f0, lds, ra1, ra1, rb);
        ktile<MF1, MF1, 3, true>(acc, f0, f1, f0, lds, ra1, ra1, rb);
      }
      ktile<MF1, MF1, 0, true>(acc, f0, f1, f0, lds, ra1, ra1, rb);
      ktile<MF1, MF1, 1, true>(acc, f0, f1, f0, lds, ra1, ra1, rb);
      ktile<MF1, MF1, 2, true>(acc, f0, f1, f0, lds, ra1, ra1, rb);
      ktile<MF1, MF2, 3, true>(acc, f0, f1, g0, lds, ra1, ra2, rb);
      MFMA_DRAIN();
      SCHED();
      if (wid == 0) STAMP(2);
      char* ob = rel == 0 ? P.out + ((size_t)(rank * MS + mh * M1 + wr * (M1 / 2)) * NS + (nt1 & 15) * 128 + wc * 64) * 2
                          : myws + WS_S + (size_t)((call & 1) * 256 + bid) * TILE1 + wid * (MF1 * 2048);
      const unsigned pitch = rel == 0 ? NS * 2 : 128;
      unsigned vo = fr * pitch + fq * 16;
#pragma unroll
      for (int mi = 0; mi < MF1; ++mi) {
        store_sys<0>(ob, vo, pack8(acc[mi][0], acc[mi][1], odd));
        store_sys<64>(ob, vo, pack8(acc[mi][2], acc[mi][3], odd));
        vo += 16 * pitch;
      }
    }
    if (rel >= 2) {
      f32x4 acc[MF2][4];
#pragma unroll
      for (int i = 0; i < MF2; ++i)
#pragma unroll
        for (int b = 0; b < 4; ++b) { acc[i][b] = (f32x4){0.f, 0.f, 0.f, 0.f}; asm volatile("" : "+v"(acc[i][b])); }
      asm volatile("s_nop 1" ::: "memory");
      SCHED();
      ktile<MF2, MF2, 0, true, FLAGK == 0>(acc, g0, g1, g0, lds, ra2, ra2, rb);
      ktile<MF2, MF2, 1, true, FLAGK == 1>(acc, g0, g1, g0, lds, ra2, ra2, rb);
      ktile<MF2, MF2, 2, true, FLAGK == 2>(acc, g0, g1, g0, lds, ra2, ra2, rb);
      ktile<MF2, MF2, 3, true, FLAGK == 3>(acc, g0, g1, g0, lds, ra2, ra2, rb);
      for (int j = 4; j < NKT - 4; j += 4) {
        ktile<MF2, MF2, 0, true>(acc, g0, g1, g0, lds, ra2, ra2, rb);
        ktile<MF2, MF2, 1, true>(acc, g0, g1, g0, lds, ra2, ra2, rb);
        ktile<MF2, MF2, 2, true>(acc, g0, g1, g0, lds, ra2, ra2, rb);
        ktile<MF2, MF2, 3, true>(acc, g0, g1, g0, lds, ra2, ra2, rb);
      }
      const unsigned tmo_v = __hip_atomic_load(tmo, __ATOMIC_RELAXED, __HIP_MEMORY_SCOPE_AGENT);
      SCHED();
      ktile<MF2, MF2, 0, true>(acc, g0, g1, g0, lds, ra2, ra2, rb);
      ktile<MF2, MF2, 1, true>(acc, g0, g1, g0, lds, ra2, ra2, rb);
      ktile<MF2, MF2, 2, true>(acc, g0, g1, g0, lds, ra2, ra2, rb);
      ktile<MF2, MF2, 3, false>(acc, g0, g1, g0, lds, ra2, ra2, rb);
      MFMA_DRAIN();
      SCHED();
      if (wid == 0) STAMP(3);
      const bool bad = tmo_v == call;
      const int blk = mt2 * 2 + wr;
      char* ob = P.out + ((size_t)((blk / 3) * MS + M1ROWS + (blk % 3) * 64) * NS + nl2 * 128 + wc * 64) * 2;
#pragma unroll
      for (int mi = 0; mi < MF2; ++mi)
#pragma unroll
        for (int b = 0; b < 2; ++b) {
          u32x4 o = pack8(acc[mi][2 * b], acc[mi][2 * b + 1], odd);
          if (bad) o = (u32x4){0x7fc07fc0u, 0x7fc07fc0u, 0x7fc07fc0u, 0x7fc07fc0u};
          store_loc(ob, vrow + mi * 16 * NS * 2 + b * 64, o);
        }
    } else {
      waitv<0>();
      BAR();
    }
  } else if (wid < 4 + NDMA) {
    const int dw = wid - 4;
    unsigned va[AP1 / NDMA], vb[16 / NDMA];
#pragma unroll
    for (int i = 0; i < AP1 / NDMA; ++i) {
      const int pc = i * NDMA + dw;
      va[i] = (unsigned)((pc * 8 + (lane >> 3)) * (KD * 2) + (((lane & 7) ^ ((4 * pc + (lane >> 4)) & 7)) * 16));
    }
#pragma unroll
    for (int i = 0; i < 16 / NDMA; ++i) {
      const int pc = i * NDMA + dw;
      vb[i] = (unsigned)((pc * 4 + (lane >> 4)) * (ND * 2) + (((lane & 15) ^ (((lane >> 4) << 2) | ((pc >> 1) & 1))) * 16));
    }
    const unsigned ldsb = (unsigned)(size_t)lds + dw * 1024;
    const char* a1 = (const char*)P.x + (size_t)(mh * M1) * KD * 2;
    const char* w1 = (const char*)P.w + (size_t)nt1 * 256;
    const char* w2 = (const char*)P.w + (size_t)(rank * 16 + nl2) * 256;
#define KX1 (mh | (((idx >> 1) & 3) << 1))
#define KX2 (m6 ^ n4)
    const int kx1 = KX1, kx2 = KX2;
#define ISSUE_B(wb, j, kt) Pieces<0, 16 / NDMA, false>::go((wb) + (size_t)(kt) * 64 * ND * 2, vb, ldsb + ((j) & 3) * 16384)
#define ISSUE1(j) do { const int kt_ = (j) ^ kx1; ISSUE_B(w1, j, kt_); \
      Pieces<0, AP1 / NDMA, false>::go(a1 + kt_ * 128, va, ldsb + A_OFF + ((j) & 3) * ASL1); } while (0)
#define ISSUE2(j) do { const int kt_ = (j) ^ kx2; ISSUE_B(w2, j, kt_); \
      Pieces<0, AP2 / NDMA / 2, true>::go(a2lo + kt_ * 128, va, ldsb + A_OFF + ((j) & 3) * ASL2); \
      Pieces<AP2 / NDMA / 2, AP2 / NDMA, true>::go(a2hi + kt_ * 128, va, ldsb + A_OFF + ((j) & 3) * ASL2); } while (0)
#pragma unroll 1
    for (int j = 0; j < 3; ++j) ISSUE1(j);
    const unsigned call = sload(callp) + 1u;
    waitv<2 * PPW1>();
    BAR();
    ISSUE1(3);
#pragma unroll 1
    for (int i = 0; i < NKT - 4; ++i) {
      waitv<2 * PPW1>(); BAR(); ISSUE1(i + 4);
    }
    if (rel >= 2) {
      const int b0 = mt2 * 2, b1 = b0 + 1;
      const char* gb = myws + WS_G + (size_t)((call & 1) * NR) * XBLK;
      const char* xb = (const char*)P.x + (size_t)M1ROWS * KD * 2;
      const char* a2lo = (b0 / 3 == rank) ? xb + (size_t)((b0 % 3) * 64) * KD * 2 : gb + (size_t)(b0 * 64) * KD * 2;
      const char* a2hi = ((b1 / 3 == rank) ? xb + (size_t)((b1 % 3) * 64) * KD * 2 : gb + (size_t)(b1 * 64) * KD * 2) - (size_t)64 * KD * 2;
      waitv<2 * PPW1>(); BAR(); ISSUE2(0);
      waitv<PPW1 + PPW2>(); BAR(); ISSUE2(1);
#pragma unroll 1
      for (int i = 2; i < NKT; ++i) {
        waitv<2 * PPW2>(); BAR(); ISSUE2(i);
      }
      waitv<2 * PPW2>(); BAR();
      waitv<PPW2>(); BAR();
      waitv<0>(); BAR();
      BAR();
    } else {
      waitv<2 * PPW1>(); BAR();
      waitv<PPW1>(); BAR();
      waitv<0>(); BAR();
      BAR();
      BAR();
    }
  } else {
    const unsigned call = sload(callp) + 1u;
    unsigned* rdy = (unsigned*)(P.ws[dst1] + WS_OF + (size_t)(rank * 32 + t32) * 128);
    if (rel >= 2) {
      const int s0 = (mt2 * 2) / 3, s1 = (mt2 * 2 + 1) / 3;
      auto gate = [&](int h, unsigned want) {
        const unsigned* c0 = (const unsigned*)(myws + WS_XC + (h * NR + s0) * 128);
        const unsigned* c1 = (const unsigned*)(myws + WS_XC + (h * NR + s1) * 128);
        const unsigned v0 = __hip_atomic_load(c0, __ATOMIC_RELAXED, __HIP_MEMORY_SCOPE_SYSTEM);
        const unsigned v1 = __hip_atomic_load(c1, __ATOMIC_RELAXED, __HIP_MEMORY_SCOPE_SYSTEM);
        SCHED();
        BAR(); BAR(); BAR(); BAR();
        SCHED();
        if (s0 != rank && rfl(v0) < want) wait_ge(c0, want, tmo, call);
        if (s1 != rank && rfl(v1) < want) wait_ge(c1, want, tmo, call);
      };
#pragma unroll 1
      for (int i = 0; i < NKT - 3 - 4; ++i) BAR();
      gate(0, call * 192u);
#pragma unroll 1
      for (int i = 0; i < 5 + FLAGK; ++i) BAR();
      if (lane == 0) __hip_atomic_store(rdy, call, __ATOMIC_RELAXED, __HIP_MEMORY_SCOPE_SYSTEM);
#pragma unroll 1
      for (int i = 0; i < 23 - FLAGK; ++i) BAR();
      gate(1, call * 32u);
      STAMP(5);
#pragma unroll 1
      for (int i = 0; i < 36; ++i) BAR();
    } else {
#pragma unroll 1
      for (int i = 0; i < NKT + 2; ++i) BAR();
      if (rel == 1 && lane == 0) __hip_atomic_store(rdy, call, __ATOMIC_RELAXED, __HIP_MEMORY_SCOPE_SYSTEM);
    }
  }

  if (rel == 1) {
    const unsigned call = sload(callp) + 1u;
    const int src = (rank + 1 + wid) & (NR - 1);
    bool ok = true;
    ok = wait_ge((const unsigned*)(myws + WS_OF + (size_t)(src * 32 + t32) * 128), call, tmo, call);
    const char* sp = P.ws[src] + WS_S + (size_t)((call & 1) * 256 + rank * 32 + t32) * TILE1;
    char* ob = P.out + ((size_t)(src * MS + mh * M1) * NS + colt(rank) * 128) * 2;
    const unsigned vl = lane * 16, vrow = (unsigned)((lane >> 3) * (NS * 2) + (lane & 7) * 16);
    if (ok) {
      u32x4 v[PDEP + 1];
#pragma unroll
      for (int i = 0; i < OPC + PDEP; ++i) {
        if (i < OPC) asm volatile("global_load_dwordx4 %0, %1, %2 sc0 sc1" : "=&v"(v[i % (PDEP + 1)]) : "v"(vl + i * 1024), "s"(sp) : "memory");
        if (i >= PDEP) {
          const int pc = i - PDEP, w = pc / (2 * MF1), r = pc % (2 * MF1);
          const int nld = (i < OPC ? i : OPC - 1) - pc, nst = pc < PDEP ? pc : PDEP;
          asm volatile("s_waitcnt vmcnt(%1)" : "+v"(v[pc % (PDEP + 1)]) : "i"(nld + nst) : "memory");
          store_loc(ob, vrow + (((w >> 1) * (M1 / 2) + r * 8) * NS + (w & 1) * 64) * 2, v[pc % (PDEP + 1)]);
        }
      }
    } else {
#pragma unroll 1
      for (int pc = 0; pc < OPC; ++pc) {
        const int w = pc / (2 * MF1), r = pc % (2 * MF1);
        store_loc(ob, vrow + (((w >> 1) * (M1 / 2) + r * 8) * NS + (w & 1) * 64) * 2,
                  (u32x4){0x7fc07fc0u, 0x7fc07fc0u, 0x7fc07fc0u, 0x7fc07fc0u});
      }
    }
    BAR();
  }
  if (wid == 6 && lane == 0) {
    __hip_atomic_fetch_add((unsigned*)(myws + WS_CALL) + bid, 1u, __ATOMIC_RELAXED, __HIP_MEMORY_SCOPE_AGENT);
  }
}

}

extern "C" void kernel_launch(int rank, int nranks, ncclComm_t comm,
                              hipStream_t stream,
                              void* const* d_in, const int* in_sizes, int n_in,
                              void* d_out, int out_size,
                              const int* in_dtypes, int out_dtype,
                              void* const* d_out_all,
                              void* const* d_ws_all, size_t ws_bytes) {
  if (nranks != NR || (unsigned)rank >= (unsigned)NR || ws_bytes < WS_NEED || n_in < 2) {
    fprintf(stderr, "unsupported configuration\n");
    return;
  }
  static bool init[NR] = {};
  if (!init[rank]) {
    HIP_CHECK(hipFuncSetAttribute((const void*)fused_gemm_a2a, hipFuncAttributeMaxDynamicSharedMemorySize, LDS_BYTES));
    init[rank] = true;
  }
  Params P;
  P.x = (const u16*)d_in[0];
  P.w = (const u16*)d_in[1];
  P.out = (char*)d_out;
  for (int p = 0; p < NR; ++p) P.ws[p] = (char*)d_ws_all[p];
  P.rank = rank;
  hipLaunchKernelGGL(fused_gemm_a2a, dim3(256), dim3(NTHREADS), LDS_BYTES, stream, P);
  HIP_CHECK(hipGetLastError());
}
```

```cpp
#include <hip/hip_runtime.h>
#include <rccl/rccl.h>
#include <cstdio>

#define HIP_CHECK(expr) do { hipError_t _e = (expr); if (_e != hipSuccess) { \
    fprintf(stderr, "HIP error %s at %s:%d\n", hipGetErrorString(_e), __FILE__, __LINE__); \
    return; } } while (0)

namespace {

typedef unsigned short u16;
typedef unsigned long long u64;
typedef unsigned u32x4 __attribute__((ext_vector_type(4)));
typedef short s16x4 __attribute__((ext_vector_type(4)));
typedef short s16x8 __attribute__((ext_vector_type(8)));
typedef __bf16 bf16x8 __attribute__((ext_vector_type(8)));
typedef float f32x4 __attribute__((ext_vector_type(4)));
#define LAS __attribute__((address_space(3)))

constexpr int NR = 8;
constexpr int MS = 512, KD = 4096, ND = 16384, NS = 2048;
constexpr int MF1 = 5, MF2 = 4;
constexpr int M1 = 32 * MF1, M2 = 32 * MF2;
constexpr int M1ROWS = 2 * M1, XROWS = MS - M1ROWS;
constexpr int NKT = KD / 64;
constexpr int NDMA = 2;
constexpr int NWAVES = 8;
constexpr int NTHREADS = NWAVES * 64;
constexpr int PA1 = M1 / 8 / NDMA, PA2 = M2 / 8 / NDMA, PB = 16 / NDMA;
constexpr int A_OFF = 65536, ASL = M1 * 128, E_OFF = A_OFF + 4 * ASL;
constexpr int LDS_BYTES = E_OFF + 16384;
static_assert(LDS_BYTES <= 163840, "lds");
constexpr int XBLK = XROWS * KD * 2;
constexpr int OPC = M1 / 4;
constexpr int TILE1 = OPC * 1024;
static_assert(XROWS == 6 * 32 && NR * XROWS == 12 * M2, "groups");

constexpr size_t WS_CALL = 0;
constexpr size_t WS_TMO = 4096;
constexpr size_t WS_XC = 8192;
constexpr size_t WS_OF = 16384;
constexpr size_t WS_STAMP = 65536;
constexpr size_t WS_G = 1u << 20;
constexpr size_t WS_S = WS_G + (size_t)2 * NR * XBLK;
constexpr size_t WS_NEED = WS_S + (size_t)2 * 256 * TILE1;

constexpr u64 DEADLINE = 100000000ull;
#define XTHR 2
#define PDEP 1
#define FLAGK 1

struct Params {
  const u16* x; const u16* w;
  char* out; char* ws[NR];
  int rank;
};

#define BAR() do { asm volatile("" ::: "memory"); __builtin_amdgcn_s_barrier(); asm volatile("" ::: "memory"); } while (0)
#define PACE1 58
#define PACE0 50
#define PACE1B PACE1
#define PACE() do { if (++pcount <= NKT + 1) { tpace += (pcount <= NKT / 2) ? PACE1 : PACE1B; while (__builtin_amdgcn_s_memrealtime() < tpace) __builtin_amdgcn_s_sleep(1); } } while (0)
#define BAR6() do { PACE(); BAR(); } while (0)
#define SCHED() __builtin_amdgcn_sched_barrier(0)
#define WAIT_LGKM(n) __builtin_amdgcn_s_waitcnt(0xC07F | ((n) << 8))
#define MFMA_DRAIN() asm volatile("s_nop 7\n\ts_nop 7\n\ts_nop 3" ::: "memory")
template <int N> __device__ __forceinline__ void waitv() { static_assert(N <= 63, "vmcnt"); asm volatile("s_waitcnt vmcnt(%0)" :: "i"(N) : "memory"); }

template <int OFF, bool SYS> __device__ __forceinline__ void glds(const char* sbase, unsigned voff, unsigned ldsbase) {
  if (SYS) asm volatile("s_add_u32 m0, %2, %3\n\ts_nop 0\n\tglobal_load_lds_dwordx4 %0, %1 sc0 sc1" :: "v"(voff), "s"(sbase), "s"(ldsbase), "i"(OFF) : "memory", "scc");
  else asm volatile("s_add_u32 m0, %2, %3\n\ts_nop 0\n\tglobal_load_lds_dwordx4 %0, %1" :: "v"(voff), "s"(sbase), "s"(ldsbase), "i"(OFF) : "memory", "scc");
}
template <int I, int N, int BASE, bool SYS> struct Pieces {
  static __device__ __forceinline__ void go(const char* sbase, const unsigned* voff, unsigned ldsbase) {
    glds<BASE + I * 1024 * NDMA, SYS>(sbase, voff[I], ldsbase);
    Pieces<I + 1, N, BASE, SYS>::go(sbase, voff, ldsbase);
  }
};
template <int N, int BASE, bool SYS> struct Pieces<N, N, BASE, SYS> { static __device__ __forceinline__ void go(const char*, const unsigned*, unsigned) {} };

__device__ __forceinline__ void store_sys(char* sbase, unsigned voff, u32x4 v) {
  asm volatile("global_store_dwordx4 %0, %1, %2 sc0 sc1\n\ts_nop 1" :: "v"(voff), "v"(v), "s"(sbase) : "memory");
}
__device__ __forceinline__ unsigned cvt_pk_bf16(float lo, float hi) {
  unsigned r; asm("v_cvt_pk_bf16_f32 %0, %1, %2" : "=v"(r) : "v"(lo), "v"(hi)); return r;
}
__device__ __forceinline__ float relu_nan(float v) { return __builtin_elementwise_maximum(v, 0.f); }
__device__ __forceinline__ unsigned rfl(unsigned v) { return (unsigned)__builtin_amdgcn_readfirstlane(v); }
__device__ __forceinline__ unsigned sload(const char* p) {
  unsigned r; asm volatile("s_load_dword %0, %1, 0x0\n\ts_waitcnt lgkmcnt(0)" : "=s"(r) : "s"(p) : "memory"); return r;
}

__device__ __forceinline__ bool wait_ge(const unsigned* flag, unsigned want, unsigned* tmo, unsigned call) {
  const u64 t0 = __builtin_amdgcn_s_memrealtime();
  while (rfl(__hip_atomic_load(flag, __ATOMIC_RELAXED, __HIP_MEMORY_SCOPE_SYSTEM)) < want) {
    if (rfl(__hip_atomic_load(tmo, __ATOMIC_RELAXED, __HIP_MEMORY_SCOPE_AGENT)) == call ||
        __builtin_amdgcn_s_memrealtime() - t0 > DEADLINE) {
      __hip_atomic_store(tmo, call, __ATOMIC_RELAXED, __HIP_MEMORY_SCOPE_AGENT);
      waitv<0>();
      return false;
    }
    __builtin_amdgcn_s_sleep(2);
  }
  return true;
}

template <int MF> struct Frag { bf16x8 a[MF]; s16x4 lo[4], hi[4]; };

template <int MF>
__device__ __forceinline__ void read_one(Frag<MF>& f, int i, LAS char* lds, int s, int h, const unsigned (&ra)[2], const unsigned (&rb)[4]) {
  if (i >= 2 && i < 2 + MF) {
    f.a[i - 2] = *(LAS bf16x8*)(lds + ra[h] + s * ASL + (i - 2) * 2048);
  } else {
    const int k = i < 2 ? i : i - MF, bn = k >> 1;
    s16x4 v = __builtin_amdgcn_ds_read_tr16_b64_v4i16((LAS s16x4*)(lds + rb[bn] + s * 16384 + h * 8192 + (k & 1) * 1024));
    if (k & 1) f.hi[bn] = v; else f.lo[bn] = v;
  }
}
template <int MF, bool RD>
__device__ __forceinline__ void half_step(f32x4 (&acc)[MF][4], const Frag<MF>& f, Frag<MF>& g, LAS char* lds, int s, int h,
                                          const unsigned (&ra)[2], const unsigned (&rb)[4]) {
#pragma unroll
  for (int i = 0; i < 4 * MF; ++i) {
    const int bn = i / MF, mi = i % MF;
    const s16x8 bv = __builtin_shufflevector(f.lo[bn], f.hi[bn], 0, 1, 2, 3, 4, 5, 6, 7);
    asm volatile("v_mfma_f32_16x16x32_bf16 %0, %1, %2, %0" : "+v"(acc[mi][bn]) : "v"(bv), "v"(f.a[mi]));
    if (RD && i < 8 + MF) read_one<MF>(g, i, lds, s, h, ra, rb);
    SCHED();
  }
}
template <int MF, int S, bool NEXT>
__device__ __forceinline__ void ktile(f32x4 (&acc)[MF][4], Frag<MF>& f0, Frag<MF>& f1, LAS char* lds,
                                      const unsigned (&ra)[2], const unsigned (&rb)[4]) {
  WAIT_LGKM(0);
  half_step<MF, true>(acc, f0, f1, lds, S, 1, ra, rb);
  WAIT_LGKM(0);
  if (S == FLAGK) waitv<0>();
  BAR();
  SCHED();
  half_step<MF, NEXT>(acc, f1, f0, lds, (S + 1) & 3, 0, ra, rb);
}
__device__ __forceinline__ u32x4 pack8(const f32x4& v0, const f32x4& v1, bool odd) {
  unsigned a0 = cvt_pk_bf16(relu_nan(v0[0]), relu_nan(v0[1])), a1 = cvt_pk_bf16(relu_nan(v0[2]), relu_nan(v0[3]));
  unsigned b0 = cvt_pk_bf16(relu_nan(v1[0]), relu_nan(v1[1])), b1 = cvt_pk_bf16(relu_nan(v1[2]), relu_nan(v1[3]));
  u32x4 o;
  o[0] = odd ? b0 : a0; o[1] = odd ? b1 : a1; o[2] = odd ? a0 : b0; o[3] = odd ? a1 : b1;
  return o;
}
template <int MF>
__device__ __forceinline__ void run_tile(LAS char* lds, int wid, int lane, char* ob, unsigned pitch, const unsigned* tmo, unsigned call) {
  const int wr = wid >> 1, wc = wid & 1, fr = lane & 15, fq = lane >> 4, q = fr >> 2, p = lane & 3;
  const unsigned rb00 = (8 * fq + q) * 256 + ((((2 * wc) ^ q) << 2) | (p ^ (fq & 1))) * 16 + (p & 1) * 8;
  const unsigned rb[4] = {rb00, rb00 ^ 8u, rb00 ^ 64u, rb00 ^ 72u};
  const unsigned rab = A_OFF + (wr * (MF * 16) + fr) * 128 + (unsigned)((fq ^ (fr >> 1)) * 16);
  const unsigned ra[2] = {rab, rab ^ 64u};
  Frag<MF> f0, f1;
#pragma unroll
  for (int i = 0; i < 8 + MF; ++i) read_one<MF>(f0, i, lds, 0, 0, ra, rb);
  f32x4 acc[MF][4];
#pragma unroll
  for (int i = 0; i < MF; ++i)
#pragma unroll
    for (int b = 0; b < 4; ++b) { acc[i][b] = (f32x4){0.f, 0.f, 0.f, 0.f}; asm volatile("" : "+v"(acc[i][b])); }
  asm volatile("s_nop 1" ::: "memory");
  SCHED();
  for (int j = 0; j < NKT - 4; j += 4) {
    ktile<MF, 0, true>(acc, f0, f1, lds, ra, rb);
    ktile<MF, 1, true>(acc, f0, f1, lds, ra, rb);
    ktile<MF, 2, true>(acc, f0, f1, lds, ra, rb);
    ktile<MF, 3, true>(acc, f0, f1, lds, ra, rb);
  }
  unsigned tmo_v = call + 1u;
  if (MF == MF2) tmo_v = __hip_atomic_load(tmo, __ATOMIC_RELAXED, __HIP_MEMORY_SCOPE_AGENT);
  SCHED();
  ktile<MF, 0, true>(acc, f0, f1, lds, ra, rb);
  ktile<MF, 1, true>(acc, f0, f1, lds, ra, rb);
  ktile<MF, 2, true>(acc, f0, f1, lds, ra, rb);
  ktile<MF, 3, false>(acc, f0, f1, lds, ra, rb);
  MFMA_DRAIN();
  SCHED();
  const bool odd = fq & 1, bad = tmo_v == call;
  LAS char* sc = lds + E_OFF + wid * 4096;
  const unsigned wo = fr * 128 + ((fq ^ (fr & 7)) * 16), ro = (lane >> 3) * 128 + (((lane & 7) ^ (lane >> 3)) * 16);
  unsigned vo = (lane >> 3) * pitch + (lane & 7) * 16;
#pragma unroll
  for (int m0 = 0; m0 < MF; m0 += 2) {
#pragma unroll
    for (int mi = m0; mi < m0 + 2 && mi < MF; ++mi) {
      *(LAS u32x4*)(sc + (mi - m0) * 2048 + wo) = pack8(acc[mi][0], acc[mi][1], odd);
      *(LAS u32x4*)(sc + (mi - m0) * 2048 + (wo ^ 64u)) = pack8(acc[mi][2], acc[mi][3], odd);
    }
    asm volatile("" ::: "memory");
#pragma unroll
    for (int r = 0; r < (m0 + 2 <= MF ? 4 : 2); ++r) {
      u32x4 o = *(LAS u32x4*)(sc + r * 1024 + ro);
      if (MF == MF2 && bad) o = (u32x4){0x7fc07fc0u, 0x7fc07fc0u, 0x7fc07fc0u, 0x7fc07fc0u};
      store_sys(ob, vo, o);
      vo += 8 * pitch;
    }
    asm volatile("" ::: "memory");
  }
}

#define STAMP(i) do {} while (0)
#define STAMPC(i) do {} while (0)

__global__ void __launch_bounds__(NTHREADS, 2) fused_gemm_a2a(const Params P) {
  extern __shared__ __attribute__((aligned(16))) char smem_[];
  LAS char* lds = (LAS char*)smem_;
  const int tid = threadIdx.x, lane = tid & 63, wid = __builtin_amdgcn_readfirstlane(tid >> 6);
  const int bid = blockIdx.x, xcd = bid & 7, idx = bid >> 3, rank = P.rank;
  const int dst1 = idx >> 2, t32 = bid & 31;
  const int rel = (dst1 - rank) & (NR - 1);
  const int mh = rel == 0 ? (xcd & 1) : (idx & 1), ct = rel == 0 ? (xcd >> 1) * 4 + (idx & 3) : ((t32 >> 4) & 1) * 8 + xcd;
  const bool hasG = rel >= 2;
  const int ci = (rel < 2 ? 0 : rel - 2) * 4 + (idx & 3), m6 = ci % 6, n4 = ci / 6;
  const int mt2 = (xcd & 1) * 6 + m6, nl2 = (xcd >> 1) * 4 + n4;
  char* myws = P.ws[rank];
  unsigned* tmo = (unsigned*)(myws + WS_TMO);
  const char* callp = myws + WS_CALL + bid * 4;
  if (wid == 0) { STAMP(0); STAMPC(8); }

  if (rel == 0 && (wid < 4 || wid >= 6)) {
    static_assert(XTHR < NR - 1, "xthr");
    const int xw = wid < 4 ? wid : wid - 2, row = t32 * 6 + xw;
    const unsigned vl = lane * 16;
    u32x4 xv[8];
    const char* xs = (const char*)P.x + (size_t)(M1ROWS + row) * KD * 2 + vl;
#pragma unroll
    for (int i = 0; i < 8; ++i) xv[i] = *(const u32x4*)(xs + i * 1024);
    const unsigned call = sload(callp) + 1u;
    const size_t goff = WS_G + (size_t)((call & 1) * NR + rank) * XBLK + (size_t)row * KD * 2;
    char* g[NR - 1];
#pragma unroll
    for (int j = 0; j < NR - 1; ++j) { int pj = j + xw; if (pj >= NR - 1) pj -= NR - 1; g[j] = P.ws[(rank + 1 + pj) & (NR - 1)] + goff; }
    char* sigp;
    {
      const int peer = (rank + 1 + lane) & (NR - 1);
      char* pw = P.ws[0];
#pragma unroll
      for (int r = 1; r < NR; ++r) pw = (peer == r) ? P.ws[r] : pw;
      sigp = pw + WS_XC + rank * 128;
    }
#pragma unroll
    for (int i = 0; i < 8; ++i) asm volatile("" : "+v"(xv[i]));
#pragma unroll
    for (int i = 0; i < 8; ++i) {
      unsigned vo = vl + i * 1024;
      asm volatile("" : "+v"(vo));
#pragma unroll
      for (int j = 0; j < NR - 1; ++j) { store_sys(g[j], vo, xv[i]); waitv<XTHR>(); }
      if (i == 4 && lane < NR - 1) __hip_atomic_fetch_add((unsigned*)sigp, 1u, __ATOMIC_RELAXED, __HIP_MEMORY_SCOPE_SYSTEM);
    }
    waitv<0>();
    if (lane < NR - 1) __hip_atomic_fetch_add((unsigned*)(sigp + NR * 128), 1u, __ATOMIC_RELAXED, __HIP_MEMORY_SCOPE_SYSTEM);
    if (wid == 0) STAMP(6);
  }
  if (wid == 7) return;

  if (wid < 4) {
    const int wr = wid >> 1, wc = wid & 1;
    const unsigned call = sload(callp) + 1u;
    BAR();
    if (wid == 0) STAMP(1);
    WAIT_LGKM(0);
    {
      char* ob = rel != 0 ? myws + WS_S + (size_t)((call & 1) * 256 + bid) * TILE1 + wid * (MF1 * 2048)
                          : P.out + ((size_t)(rank * MS + mh * M1 + wr * (M1 / 2)) * NS + ct * 128 + wc * 64) * 2;
      run_tile<MF1>(lds, wid, lane, ob, rel != 0 ? 128u : (unsigned)(NS * 2), tmo, call);
      if (wid == 0) { STAMP(rel == 0 ? 3 : 2); WAIT_LGKM(0); }
    }
    if (hasG) {
      const int blk = mt2 * 2 + wr;
      char* ob = P.out + ((size_t)((blk / 3) * MS + M1ROWS + (blk % 3) * 64) * NS + nl2 * 128 + wc * 64) * 2;
      run_tile<MF2>(lds, wid, lane, ob, NS * 2, tmo, call);
      if (wid == 0) STAMP(3);
    }
    if (rel == 1) {
      waitv<0>();
      BAR();
    }
  } else if (wid < 4 + NDMA) {
    const int dw = wid - 4;
    unsigned va[PA1], vb[PB];
#pragma unroll
    for (int i = 0; i < PA1; ++i) {
      const int pc = i * NDMA + dw;
      va[i] = (unsigned)((pc * 8 + (lane >> 3)) * (KD * 2) + (((lane & 7) ^ ((4 * pc + (lane >> 4)) & 7)) * 16));
    }
#pragma unroll
    for (int i = 0; i < PB; ++i) {
      const int pc = i * NDMA + dw;
      vb[i] = (unsigned)((pc * 4 + (lane >> 4)) * (ND * 2) + (((lane & 15) ^ (((lane >> 4) << 2) | ((pc >> 1) & 1))) * 16));
    }
    const unsigned ldsb = (unsigned)(size_t)lds + dw * 1024;
    const char* a1 = (const char*)P.x + (size_t)(mh * M1) * KD * 2;
#define KX1 (mh | (((idx >> 1) & 3) << 1))
#define KX2 ((m6 ^ n4) | ((xcd & 3) << 3))
#define LROT 48
    const int kx1 = rel == 0 ? ((idx & 3) | ((xcd & 3) << 3)) : (KX1), kr1 = rel == 0 ? LROT : 0, kx2 = KX2;
#define LD_B(wb, kt, S) Pieces<0, PB, (S) * 16384, false>::go((wb) + (size_t)(kt) * 64 * ND * 2, vb, ldsb)
#define KT1(j) ((((j) + kr1) & (NKT - 1)) ^ kx1)
#define LD_1(j, S) do { const int kt_ = KT1(j); LD_B(w1, kt_, S); Pieces<0, PA1, A_OFF + (S) * ASL, false>::go(a1 + kt_ * 128, va, ldsb); } while (0)
#define LD_A2(j, S) do { Pieces<0, PA2 / 2, A_OFF + (S) * ASL, true>::go(a2lo + ((j) ^ kx2) * 128, va, ldsb); \
                         Pieces<PA2 / 2, PA2, A_OFF + (S) * ASL, true>::go(a2hi + ((j) ^ kx2) * 128, va, ldsb); } while (0)
#define QUAD2(j, W0, W1, W2, W3) do { \
      waitv<W0>(); BAR(); LD_B(w2, (j) ^ kx2, 0); LD_A2(j, 0); \
      waitv<W1>(); BAR(); LD_B(w2, ((j) + 1) ^ kx2, 1); LD_A2((j) + 1, 1); \
      waitv<W2>(); BAR(); LD_B(w2, ((j) + 2) ^ kx2, 2); LD_A2((j) + 2, 2); \
      waitv<W3>(); BAR(); LD_B(w2, ((j) + 3) ^ kx2, 3); LD_A2((j) + 3, 3); } while (0)
    constexpr int P1 = PA1 + PB, P2 = PA2 + PB;
    {
      const char* w1 = (const char*)P.w + (size_t)(dst1 * 16 + ct) * 256;
      LD_1(0, 0); LD_1(1, 1); LD_1(2, 2);
      waitv<2 * P1>();
      if (rel == 0) { LD_1(3, 3); BAR(); } else { BAR(); LD_1(3, 3); }
#pragma unroll 1
      for (int j = 4; j < NKT; j += 4) {
        waitv<2 * P1>(); BAR(); LD_1(j, 0);
        waitv<2 * P1>(); BAR(); LD_1(j + 1, 1);
        waitv<2 * P1>(); BAR(); LD_1(j + 2, 2);
        waitv<2 * P1>(); BAR(); LD_1(j + 3, 3);
      }
    }
    if (hasG) {
      const unsigned call = sload(callp) + 1u;
      const char* w2 = (const char*)P.w + (size_t)(rank * 16 + nl2) * 256;
      const int b0 = mt2 * 2, b1 = b0 + 1;
      const char* gb = myws + WS_G + (size_t)((call & 1) * NR) * XBLK;
      const char* xb = (const char*)P.x + (size_t)M1ROWS * KD * 2;
      const char* a2lo = (b0 / 3 == rank) ? xb + (size_t)((b0 % 3) * 64) * KD * 2 : gb + (size_t)(b0 * 64) * KD * 2;
      const char* a2hi = ((b1 / 3 == rank) ? xb + (size_t)((b1 % 3) * 64) * KD * 2 : gb + (size_t)(b1 * 64) * KD * 2) - (size_t)64 * KD * 2;
      QUAD2(0, 2 * P1, P1 + P2, 2 * P2, 2 * P2);
#pragma unroll 1
      for (int j = 4; j < NKT; j += 4) QUAD2(j, 2 * P2, 2 * P2, 2 * P2, 2 * P2);
      waitv<2 * P2>(); BAR();
      waitv<P2>(); BAR();
    } else {
      waitv<2 * P1>(); BAR();
      waitv<P1>(); BAR();
    }
    waitv<0>(); BAR();
    BAR();
    if (rel == 1) BAR();
  } else {
    int tcount = -1; (void)tcount;
    int pcount = rel == 0 ? 1000 : 0; u64 tpace = __builtin_amdgcn_s_memrealtime() + PACE0;
    const unsigned call = sload(callp) + 1u;
    unsigned* rdy = (unsigned*)(P.ws[dst1] + WS_OF + (size_t)(rank * 32 + t32) * 128);
    const int s0 = (mt2 * 2) / 3, s1 = (mt2 * 2 + 1) / 3;
    auto gate = [&](int h, int nb) {
      const unsigned* c0 = (const unsigned*)(myws + WS_XC + (h * NR + s0) * 128);
      const unsigned* c1 = (const unsigned*)(myws + WS_XC + (h * NR + s1) * 128);
      const unsigned v0 = __hip_atomic_load(c0, __ATOMIC_RELAXED, __HIP_MEMORY_SCOPE_SYSTEM);
      const unsigned v1 = __hip_atomic_load(c1, __ATOMIC_RELAXED, __HIP_MEMORY_SCOPE_SYSTEM);
      SCHED();
#pragma unroll 1
      for (int i = 0; i < nb; ++i) BAR6();
      SCHED();
      if (s0 != rank && rfl(v0) < call * 192u) wait_ge(c0, call * 192u, tmo, call);
      if (s1 != rank && rfl(v1) < call * 192u) wait_ge(c1, call * 192u, tmo, call);
    };
    auto bars = [&](int n) {
#pragma unroll 1
      for (int i = 0; i < n; ++i) BAR6();
    };
    if (rel == 0) {
      bars(NKT + 1);
    } else if (rel == 1) {
      bars(NKT + 2);
      if (lane == 0) __hip_atomic_store(rdy, call, __ATOMIC_RELAXED, __HIP_MEMORY_SCOPE_SYSTEM);
    } else {
      bars(NKT - 3 - 4);
      gate(0, 4);
      bars(5 + FLAGK);
      if (lane == 0) __hip_atomic_store(rdy, call, __ATOMIC_RELAXED, __HIP_MEMORY_SCOPE_SYSTEM);
      bars(23 - FLAGK);
      gate(1, 4);
      STAMP(5);
      bars(36);
    }
  }

  if (rel == 1) {
    const unsigned call = sload(callp) + 1u;
    const int src = (rank + 1 + wid) & (NR - 1);
    bool ok = true;
    ok = wait_ge((const unsigned*)(myws + WS_OF + (size_t)(src * 32 + t32) * 128), call, tmo, call);
    const char* sp = P.ws[src] + WS_S + (size_t)((call & 1) * 256 + rank * 32 + t32) * TILE1;
    char* ob = P.out + ((size_t)(src * MS + mh * M1) * NS + ct * 128) * 2;
    const unsigned vl = lane * 16, vrow = (unsigned)((lane >> 3) * (NS * 2) + (lane & 7) * 16);
    if (ok) {
      u32x4 v[PDEP + 1];
#pragma unroll
      for (int i = 0; i < OPC + PDEP; ++i) {
        if (i < OPC) asm volatile("global_load_dwordx4 %0, %1, %2 sc0 sc1" : "=&v"(v[i % (PDEP + 1)]) : "v"(vl + i * 1024), "s"(sp) : "memory");
        if (i >= PDEP) {
          const int pc = i - PDEP, w = pc / (2 * MF1), r = pc % (2 * MF1);
          const int nld = (i < OPC ? i : OPC - 1) - pc, nst = pc < PDEP ? pc : PDEP;
          asm volatile("s_waitcnt vmcnt(%1)" : "+v"(v[pc % (PDEP + 1)]) : "i"(nld + nst) : "memory");
          store_sys(ob, vrow + (((w >> 1) * (M1 / 2) + r * 8) * NS + (w & 1) * 64) * 2, v[pc % (PDEP + 1)]);
        }
      }
    } else {
#pragma unroll 1
      for (int pc = 0; pc < OPC; ++pc) {
        const int w = pc / (2 * MF1), r = pc % (2 * MF1);
        store_sys(ob, vrow + (((w >> 1) * (M1 / 2) + r * 8) * NS + (w & 1) * 64) * 2,
                  (u32x4){0x7fc07fc0u, 0x7fc07fc0u, 0x7fc07fc0u, 0x7fc07fc0u});
      }
    }
    BAR();
  }
  if (wid == 6 && lane == 0) {
    __hip_atomic_fetch_add((unsigned*)(myws + WS_CALL) + bid, 1u, __ATOMIC_RELAXED, __HIP_MEMORY_SCOPE_AGENT);
  }
}

}

extern "C" void kernel_launch(int rank, int nranks, ncclComm_t comm,
                              hipStream_t stream,
                              void* const* d_in, const int* in_sizes, int n_in,
                              void* d_out, int out_size,
                              const int* in_dtypes, int out_dtype,
                              void* const* d_out_all,
                              void* const* d_ws_all, size_t ws_bytes) {
  if (nranks != NR || (unsigned)rank >= (unsigned)NR || ws_bytes < WS_NEED || n_in < 2) {
    fprintf(stderr, "unsupported configuration\n");
    return;
  }
  static bool init[NR] = {};
  if (!init[rank]) {
    HIP_CHECK(hipFuncSetAttribute((const void*)fused_gemm_a2a, hipFuncAttributeMaxDynamicSharedMemorySize, LDS_BYTES));
    init[rank] = true;
  }
  Params P;
  P.x = (const u16*)d_in[0];
  P.w = (const u16*)d_in[1];
  P.out = (char*)d_out;
  for (int p = 0; p < NR; ++p) P.ws[p] = (char*)d_ws_all[p];
  P.rank = rank;
  hipLaunchKernelGGL(fused_gemm_a2a, dim3(256), dim3(NTHREADS), LDS_BYTES, stream, P);
  HIP_CHECK(hipGetLastError());
}
```

```cpp
#include <hip/hip_runtime.h>
#include <rccl/rccl.h>
#include <cstdio>

#define HIP_CHECK(expr) do { hipError_t _e = (expr); if (_e != hipSuccess) { \
    fprintf(stderr, "HIP error %s at %s:%d\n", hipGetErrorString(_e), __FILE__, __LINE__); \
    return; } } while (0)

namespace {

typedef unsigned short u16;
typedef unsigned long long u64;
typedef unsigned u32x4 __attribute__((ext_vector_type(4)));
typedef short s16x4 __attribute__((ext_vector_type(4)));
typedef short s16x8 __attribute__((ext_vector_type(8)));
typedef __bf16 bf16x8 __attribute__((ext_vector_type(8)));
typedef float f32x4 __attribute__((ext_vector_type(4)));
#define LAS __attribute__((address_space(3)))

constexpr int NR = 8;
constexpr int MS = 512, KD = 4096, ND = 16384, NS = 2048;
constexpr int MF1 = 5, MF2 = 4;
constexpr int M1 = 32 * MF1, M2 = 32 * MF2;
constexpr int M1ROWS = 2 * M1, XROWS = MS - M1ROWS;
constexpr int NKT = KD / 64;
constexpr int NDMA = 2;
constexpr int NWAVES = 8;
constexpr int NTHREADS = NWAVES * 64;
constexpr int PA1 = M1 / 8 / NDMA, PA2 = M2 / 8 / NDMA, PB = 16 / NDMA;
constexpr int A_OFF = 65536, ASL = M1 * 128, E_OFF = A_OFF + 4 * ASL;
constexpr int LDS_BYTES = E_OFF + 16384;
static_assert(LDS_BYTES <= 163840, "lds");
constexpr int XBLK = XROWS * KD * 2;
constexpr int OPC = M1 / 4;
constexpr int TILE1 = OPC * 1024;
static_assert(XROWS == 6 * 32 && NR * XROWS == 12 * M2, "groups");

constexpr size_t WS_CALL = 0;
constexpr size_t WS_TMO = 4096;
constexpr size_t WS_XC = 8192;
constexpr size_t WS_OF = 16384;
constexpr size_t WS_STAMP = 65536;
constexpr size_t WS_G = 1u << 20;
constexpr size_t WS_S = WS_G + (size_t)2 * NR * XBLK;
constexpr size_t WS_NEED = WS_S + (size_t)2 * 256 * TILE1;

constexpr u64 DEADLINE = 100000000ull;
#define XTHR 2
#define PDEP 1
#define FLAGK 1

#define LSTART 3650

struct Params {
  const u16* x; const u16* w;
  char* out; char* ws[NR];
  int rank;
};

#define BAR() do { asm volatile("" ::: "memory"); __builtin_amdgcn_s_barrier(); asm volatile("" ::: "memory"); } while (0)
#define PACE1 55
#define PACE0 50
#define PACE1B PACE1
#define PACE() do { if (++pcount <= NKT + 1) { tpace += (pcount <= NKT / 2) ? PACE1 : PACE1B; while (__builtin_amdgcn_s_memrealtime() < tpace) __builtin_amdgcn_s_sleep(1); } } while (0)
#define BAR6() do { PACE(); BAR(); } while (0)
#define SCHED() __builtin_amdgcn_sched_barrier(0)
#define WAIT_LGKM(n) __builtin_amdgcn_s_waitcnt(0xC07F | ((n) << 8))
#define MFMA_DRAIN() asm volatile("s_nop 7\n\ts_nop 7\n\ts_nop 3" ::: "memory")
template <int N> __device__ __forceinline__ void waitv() { static_assert(N <= 63, "vmcnt"); asm volatile("s_waitcnt vmcnt(%0)" :: "i"(N) : "memory"); }

template <int OFF, int SYS> __device__ __forceinline__ void glds(const char* sbase, unsigned voff, unsigned ldsbase) {
#define WFL " nt"
  if (SYS == 2) asm volatile("s_add_u32 m0, %2, %3\n\ts_nop 0\n\tglobal_load_lds_dwordx4 %0, %1" WFL :: "v"(voff), "s"(sbase), "s"(ldsbase), "i"(OFF) : "memory", "scc");
  else if (SYS == 1) asm volatile("s_add_u32 m0, %2, %3\n\ts_nop 0\n\tglobal_load_lds_dwordx4 %0, %1 sc0 sc1" :: "v"(voff), "s"(sbase), "s"(ldsbase), "i"(OFF) : "memory", "scc");
  else asm volatile("s_add_u32 m0, %2, %3\n\ts_nop 0\n\tglobal_load_lds_dwordx4 %0, %1" :: "v"(voff), "s"(sbase), "s"(ldsbase), "i"(OFF) : "memory", "scc");
}
template <int I, int N, int BASE, int SYS> struct Pieces {
  static __device__ __forceinline__ void go(const char* sbase, const unsigned* voff, unsigned ldsbase) {
    glds<BASE + I * 1024 * NDMA, SYS>(sbase, voff[I], ldsbase);
    Pieces<I + 1, N, BASE, SYS>::go(sbase, voff, ldsbase);
  }
};
template <int N, int BASE, int SYS> struct Pieces<N, N, BASE, SYS> { static __device__ __forceinline__ void go(const char*, const unsigned*, unsigned) {} };

__device__ __forceinline__ void store_sys(char* sbase, unsigned voff, u32x4 v) {
  asm volatile("global_store_dwordx4 %0, %1, %2 sc0 sc1\n\ts_nop 1" :: "v"(voff), "v"(v), "s"(sbase) : "memory");
}
__device__ __forceinline__ unsigned cvt_pk_bf16(float lo, float hi) {
  unsigned r; asm("v_cvt_pk_bf16_f32 %0, %1, %2" : "=v"(r) : "v"(lo), "v"(hi)); return r;
}
__device__ __forceinline__ float relu_nan(float v) { return __builtin_elementwise_maximum(v, 0.f); }
__device__ __forceinline__ unsigned rfl(unsigned v) { return (unsigned)__builtin_amdgcn_readfirstlane(v); }
__device__ __forceinline__ unsigned sload(const char* p) {
  unsigned r; asm volatile("s_load_dword %0, %1, 0x0\n\ts_waitcnt lgkmcnt(0)" : "=s"(r) : "s"(p) : "memory"); return r;
}

__device__ __forceinline__ bool wait_ge(const unsigned* flag, unsigned want, unsigned* tmo, unsigned call) {
  const u64 t0 = __builtin_amdgcn_s_memrealtime();
  while (rfl(__hip_atomic_load(flag, __ATOMIC_RELAXED, __HIP_MEMORY_SCOPE_SYSTEM)) < want) {
    if (rfl(__hip_atomic_load(tmo, __ATOMIC_RELAXED, __HIP_MEMORY_SCOPE_AGENT)) == call ||
        __builtin_amdgcn_s_memrealtime() - t0 > DEADLINE) {
      __hip_atomic_store(tmo, call, __ATOMIC_RELAXED, __HIP_MEMORY_SCOPE_AGENT);
      waitv<0>();
      return false;
    }
    __builtin_amdgcn_s_sleep(2);
  }
  return true;
}

template <int MF> struct Frag { bf16x8 a[MF]; s16x4 lo[4], hi[4]; };

template <int MF>
__device__ __forceinline__ void read_one(Frag<MF>& f, int i, LAS char* lds, int s, int h, const unsigned (&ra)[2], const unsigned (&rb)[4]) {
  if (i >= 2 && i < 2 + MF) {
    f.a[i - 2] = *(LAS bf16x8*)(lds + ra[h] + s * ASL + (i - 2) * 2048);
  } else {
    const int k = i < 2 ? i : i - MF, bn = k >> 1;
    s16x4 v = __builtin_amdgcn_ds_read_tr16_b64_v4i16((LAS s16x4*)(lds + rb[bn] + s * 16384 + h * 8192 + (k & 1) * 1024));
    if (k & 1) f.hi[bn] = v; else f.lo[bn] = v;
  }
}
template <int MF, bool RD>
__device__ __forceinline__ void half_step(f32x4 (&acc)[MF][4], const Frag<MF>& f, Frag<MF>& g, LAS char* lds, int s, int h,
                                          const unsigned (&ra)[2], const unsigned (&rb)[4]) {
#pragma unroll
  for (int i = 0; i < 4 * MF; ++i) {
    const int bn = i / MF, mi = (bn & 1) ? MF - 1 - i % MF : i % MF;
    const s16x8 bv = __builtin_shufflevector(f.lo[bn], f.hi[bn], 0, 1, 2, 3, 4, 5, 6, 7);
    asm volatile("v_mfma_f32_16x16x32_bf16 %0, %1, %2, %0" : "+v"(acc[mi][bn]) : "v"(bv), "v"(f.a[mi]));
    if (RD && i < 8 + MF) read_one<MF>(g, i, lds, s, h, ra, rb);
    SCHED();
  }
}
template <int MF, int S, bool NEXT>
__device__ __forceinline__ void ktile(f32x4 (&acc)[MF][4], Frag<MF>& f0, Frag<MF>& f1, LAS char* lds,
                                      const unsigned (&ra)[2], const unsigned (&rb)[4]) {
  WAIT_LGKM(0);
  half_step<MF, true>(acc, f0, f1, lds, S, 1, ra, rb);
  WAIT_LGKM(0);
  if (S == FLAGK) waitv<0>();
  BAR();
  SCHED();
  half_step<MF, NEXT>(acc, f1, f0, lds, (S + 1) & 3, 0, ra, rb);
}
__device__ __forceinline__ u32x4 pack8(const f32x4& v0, const f32x4& v1, bool odd) {
  unsigned a0 = cvt_pk_bf16(relu_nan(v0[0]), relu_nan(v0[1])), a1 = cvt_pk_bf16(relu_nan(v0[2]), relu_nan(v0[3]));
  unsigned b0 = cvt_pk_bf16(relu_nan(v1[0]), relu_nan(v1[1])), b1 = cvt_pk_bf16(relu_nan(v1[2]), relu_nan(v1[3]));
  u32x4 o;
  o[0] = odd ? b0 : a0; o[1] = odd ? b1 : a1; o[2] = odd ? a0 : b0; o[3] = odd ? a1 : b1;
  return o;
}
template <int MF>
__device__ __forceinline__ void run_tile(LAS char* lds, int wid, int lane, char* ob, unsigned pitch, const unsigned* tmo, unsigned call) {
  const int wr = wid >> 1, wc = wid & 1, fr = lane & 15, fq = lane >> 4, q = fr >> 2, p = lane & 3;
  const unsigned rb00 = (8 * fq + q) * 256 + ((((2 * wc) ^ q) << 2) | (p ^ (fq & 1))) * 16 + (p & 1) * 8;
  const unsigned rb[4] = {rb00, rb00 ^ 8u, rb00 ^ 64u, rb00 ^ 72u};
  const unsigned rab = A_OFF + (wr * (MF * 16) + fr) * 128 + (unsigned)((fq ^ (fr >> 1)) * 16);
  const unsigned ra[2] = {rab, rab ^ 64u};
  Frag<MF> f0, f1;
#pragma unroll
  for (int i = 0; i < 8 + MF; ++i) read_one<MF>(f0, i, lds, 0, 0, ra, rb);
  f32x4 acc[MF][4];
#pragma unroll
  for (int i = 0; i < MF; ++i)
#pragma unroll
    for (int b = 0; b < 4; ++b) { acc[i][b] = (f32x4){0.f, 0.f, 0.f, 0.f}; asm volatile("" : "+v"(acc[i][b])); }
  asm volatile("s_nop 1" ::: "memory");
  SCHED();
  for (int j = 0; j < NKT - 4; j += 4) {
    ktile<MF, 0, true>(acc, f0, f1, lds, ra, rb);
    ktile<MF, 1, true>(acc, f0, f1, lds, ra, rb);
    ktile<MF, 2, true>(acc, f0, f1, lds, ra, rb);
    ktile<MF, 3, true>(acc, f0, f1, lds, ra, rb);
  }
  unsigned tmo_v = call + 1u;
  if (MF == MF2) tmo_v = __hip_atomic_load(tmo, __ATOMIC_RELAXED, __HIP_MEMORY_SCOPE_AGENT);
  SCHED();
  ktile<MF, 0, true>(acc, f0, f1, lds, ra, rb);
  ktile<MF, 1, true>(acc, f0, f1, lds, ra, rb);
  ktile<MF, 2, true>(acc, f0, f1, lds, ra, rb);
  ktile<MF, 3, false>(acc, f0, f1, lds, ra, rb);
  MFMA_DRAIN();
  SCHED();
  const bool odd = fq & 1, bad = tmo_v == call;
  LAS char* sc = lds + E_OFF + wid * 4096;
  const unsigned wo = fr * 128 + ((fq ^ (fr & 7)) * 16), ro = (lane >> 3) * 128 + (((lane & 7) ^ (lane >> 3)) * 16);
  unsigned vo = (lane >> 3) * pitch + (lane & 7) * 16;
#pragma unroll
  for (int m0 = 0; m0 < MF; m0 += 2) {
#pragma unroll
    for (int mi = m0; mi < m0 + 2 && mi < MF; ++mi) {
      *(LAS u32x4*)(sc + (mi - m0) * 2048 + wo) = pack8(acc[mi][0], acc[mi][1], odd);
      *(LAS u32x4*)(sc + (mi - m0) * 2048 + (wo ^ 64u)) = pack8(acc[mi][2], acc[mi][3], odd);
    }
    asm volatile("" ::: "memory");
#pragma unroll
    for (int r = 0; r < (m0 + 2 <= MF ? 4 : 2); ++r) {
      u32x4 o = *(LAS u32x4*)(sc + r * 1024 + ro);
      if (MF == MF2 && bad) o = (u32x4){0x7fc07fc0u, 0x7fc07fc0u, 0x7fc07fc0u, 0x7fc07fc0u};
      store_sys(ob, vo, o);
      vo += 8 * pitch;
    }
    asm volatile("" ::: "memory");
  }
}

#define STAMP(i) do {} while (0)
#define STAMPC(i) do {} while (0)

__global__ void __launch_bounds__(NTHREADS, 2) fused_gemm_a2a(const Params P) {
  extern __shared__ __attribute__((aligned(16))) char smem_[];
  LAS char* lds = (LAS char*)smem_;
  const int tid = threadIdx.x, lane = tid & 63, wid = __builtin_amdgcn_readfirstlane(tid >> 6);
  const int bid = blockIdx.x, xcd = bid & 7, idx = bid >> 3, rank = P.rank;
  const int dst1 = idx >> 2, t32 = bid & 31;
  const int rel = (dst1 - rank) & (NR - 1);
  const int mh = rel == 0 ? (xcd & 1) : (idx & 1), ct = rel == 0 ? (xcd >> 1) * 4 + (idx & 3) : ((t32 >> 4) & 1) * 8 + xcd;
  const bool hasG = rel >= 2;
  const int ci = (rel < 2 ? 0 : rel - 2) * 4 + (idx & 3), m6 = ci % 6, n4 = ci / 6;
  const int mt2 = (xcd & 1) * 6 + m6, nl2 = (xcd >> 1) * 4 + n4;
  char* myws = P.ws[rank];
  unsigned* tmo = (unsigned*)(myws + WS_TMO);
  const char* callp = myws + WS_CALL + bid * 4;
  if (wid == 0) { STAMP(0); STAMPC(8); }
  const u64 tk0 = __builtin_amdgcn_s_memrealtime();

  if (rel == 0 && (wid < 4 || wid >= 6)) {
    static_assert(XTHR < NR - 1, "xthr");
    const int xw = wid < 4 ? wid : wid - 2, row = t32 * 6 + xw;
    const unsigned vl = lane * 16;
    u32x4 xv[8];
    const char* xs = (const char*)P.x + (size_t)(M1ROWS + row) * KD * 2 + vl;
#pragma unroll
    for (int i = 0; i < 8; ++i) xv[i] = *(const u32x4*)(xs + i * 1024);
    const unsigned call = sload(callp) + 1u;
    const size_t goff = WS_G + (size_t)((call & 1) * NR + rank) * XBLK + (size_t)row * KD * 2;
    char* g[NR - 1];
#pragma unroll
    for (int j = 0; j < NR - 1; ++j) { int pj = j + xw; if (pj >= NR - 1) pj -= NR - 1; g[j] = P.ws[(rank + 1 + pj) & (NR - 1)] + goff; }
    char* sigp;
    {
      const int peer = (rank + 1 + lane) & (NR - 1);
      char* pw = P.ws[0];
#pragma unroll
      for (int r = 1; r < NR; ++r) pw = (peer == r) ? P.ws[r] : pw;
      sigp = pw + WS_XC + rank * 128;
    }
#pragma unroll
    for (int i = 0; i < 8; ++i) asm volatile("" : "+v"(xv[i]));
#pragma unroll
    for (int i = 0; i < 8; ++i) {
      unsigned vo = vl + i * 1024;
      asm volatile("" : "+v"(vo));
#pragma unroll
      for (int j = 0; j < NR - 1; ++j) { store_sys(g[j], vo, xv[i]); waitv<XTHR>(); }
      if (i == 4 && lane < NR - 1) __hip_atomic_fetch_add((unsigned*)sigp, 1u, __ATOMIC_RELAXED, __HIP_MEMORY_SCOPE_SYSTEM);
    }
    waitv<0>();
    if (lane < NR - 1) __hip_atomic_fetch_add((unsigned*)(sigp + NR * 128), 1u, __ATOMIC_RELAXED, __HIP_MEMORY_SCOPE_SYSTEM);
    if (wid == 0) STAMP(6);
  }
  if (wid == 7) return;

  if (wid < 4) {
    const int wr = wid >> 1, wc = wid & 1;
    const unsigned call = sload(callp) + 1u;
    BAR();
    if (wid == 0) STAMP(1);
    WAIT_LGKM(0);
    {
      char* ob = rel != 0 ? myws + WS_S + (size_t)((call & 1) * 256 + bid) * TILE1 + wid * (MF1 * 2048)
                          : P.out + ((size_t)(rank * MS + mh * M1 + wr * (M1 / 2)) * NS + ct * 128 + wc * 64) * 2;
      run_tile<MF1>(lds, wid, lane, ob, rel != 0 ? 128u : (unsigned)(NS * 2), tmo, call);
      if (wid == 0) { STAMP(rel == 0 ? 3 : 2); WAIT_LGKM(0); }
    }
    if (hasG) {
      const int blk = mt2 * 2 + wr;
      char* ob = P.out + ((size_t)((blk / 3) * MS + M1ROWS + (blk % 3) * 64) * NS + nl2 * 128 + wc * 64) * 2;
      run_tile<MF2>(lds, wid, lane, ob, NS * 2, tmo, call);
      if (wid == 0) STAMP(3);
    }
    if (rel == 1) {
      waitv<0>();
      BAR();
    }
  } else if (wid < 4 + NDMA) {
    const int dw = wid - 4;
    unsigned va[PA1], vb[PB];
#pragma unroll
    for (int i = 0; i < PA1; ++i) {
      const int pc = i * NDMA + dw;
      va[i] = (unsigned)((pc * 8 + (lane >> 3)) * (KD * 2) + (((lane & 7) ^ ((4 * pc + (lane >> 4)) & 7)) * 16));
    }
#pragma unroll
    for (int i = 0; i < PB; ++i) {
      const int pc = i * NDMA + dw;
      vb[i] = (unsigned)((pc * 4 + (lane >> 4)) * (ND * 2) + (((lane & 15) ^ (((lane >> 4) << 2) | ((pc >> 1) & 1))) * 16));
    }
    const unsigned ldsb = (unsigned)(size_t)lds + dw * 1024;
    const char* a1 = (const char*)P.x + (size_t)(mh * M1) * KD * 2;
#define KX1 (mh | (((idx >> 1) & 3) << 1))
#define KX2 ((m6 ^ n4) | ((xcd & 3) << 3))
#define LROT 0
    const int kx1 = rel == 0 ? ((idx & 3) | ((xcd & 3) << 3)) : (KX1), kr1 = rel == 0 ? LROT : 0, kx2 = KX2;
#define LD_B(wb, kt, S) Pieces<0, PB, (S) * 16384, false>::go((wb) + (size_t)(kt) * 64 * ND * 2, vb, ldsb)
#define KT1(j) ((((j) + kr1) & (NKT - 1)) ^ kx1)
#define LD_B1 LD_B
#define LD_1(j, S) do { const int kt_ = KT1(j); LD_B1(w1, kt_, S); Pieces<0, PA1, A_OFF + (S) * ASL, false>::go(a1 + kt_ * 128, va, ldsb); } while (0)
#define LD_A2(j, S) do { Pieces<0, PA2 / 2, A_OFF + (S) * ASL, true>::go(a2lo + ((j) ^ kx2) * 128, va, ldsb); \
                         Pieces<PA2 / 2, PA2, A_OFF + (S) * ASL, true>::go(a2hi + ((j) ^ kx2) * 128, va, ldsb); } while (0)
#define QUAD2(j, W0, W1, W2, W3) do { \
      waitv<W0>(); BAR(); LD_B(w2, (j) ^ kx2, 0); LD_A2(j, 0); \
      waitv<W1>(); BAR(); LD_B(w2, ((j) + 1) ^ kx2, 1); LD_A2((j) + 1, 1); \
      waitv<W2>(); BAR(); LD_B(w2, ((j) + 2) ^ kx2, 2); LD_A2((j) + 2, 2); \
      waitv<W3>(); BAR(); LD_B(w2, ((j) + 3) ^ kx2, 3); LD_A2((j) + 3, 3); } while (0)
    constexpr int P1 = PA1 + PB, P2 = PA2 + PB;
    {
      const char* w1 = (const char*)P.w + (size_t)(dst1 * 16 + ct) * 256;
      LD_1(0, 0); LD_1(1, 1); LD_1(2, 2);
      waitv<2 * P1>();
      if (rel == 0) { LD_1(3, 3); BAR(); } else { BAR(); LD_1(3, 3); }
#pragma unroll 1
      for (int j = 4; j < NKT; j += 4) {
        waitv<2 * P1>(); BAR(); LD_1(j, 0);
        waitv<2 * P1>(); BAR(); LD_1(j + 1, 1);
        waitv<2 * P1>(); BAR(); LD_1(j + 2, 2);
        waitv<2 * P1>(); BAR(); LD_1(j + 3, 3);
      }
    }
    if (hasG) {
      const unsigned call = sload(callp) + 1u;
      const char* w2 = (const char*)P.w + (size_t)(rank * 16 + nl2) * 256;
      const int b0 = mt2 * 2, b1 = b0 + 1;
      const char* gb = myws + WS_G + (size_t)((call & 1) * NR) * XBLK;
      const char* xb = (const char*)P.x + (size_t)M1ROWS * KD * 2;
      const char* a2lo = (b0 / 3 == rank) ? xb + (size_t)((b0 % 3) * 64) * KD * 2 : gb + (size_t)(b0 * 64) * KD * 2;
      const char* a2hi = ((b1 / 3 == rank) ? xb + (size_t)((b1 % 3) * 64) * KD * 2 : gb + (size_t)(b1 * 64) * KD * 2) - (size_t)64 * KD * 2;
      QUAD2(0, 2 * P1, P1 + P2, 2 * P2, 2 * P2);
#pragma unroll 1
      for (int j = 4; j < NKT; j += 4) QUAD2(j, 2 * P2, 2 * P2, 2 * P2, 2 * P2);
      waitv<2 * P2>(); BAR();
      waitv<P2>(); BAR();
    } else {
      waitv<2 * P1>(); BAR();
      waitv<P1>(); BAR();
    }
    waitv<0>(); BAR();
    BAR();
    if (rel == 1) BAR();
  } else {
    int tcount = -1; (void)tcount;
    int pcount = rel == 0 ? 1000 : 0; u64 tpace = __builtin_amdgcn_s_memrealtime() + PACE0;
    const unsigned call = sload(callp) + 1u;
    unsigned* rdy = (unsigned*)(P.ws[dst1] + WS_OF + (size_t)(rank * 32 + t32) * 128);
    const int s0 = (mt2 * 2) / 3, s1 = (mt2 * 2 + 1) / 3;
    auto gate = [&](int h, int nb) {
      const unsigned* c0 = (const unsigned*)(myws + WS_XC + (h * NR + s0) * 128);
      const unsigned* c1 = (const unsigned*)(myws + WS_XC + (h * NR + s1) * 128);
      const unsigned v0 = __hip_atomic_load(c0, __ATOMIC_RELAXED, __HIP_MEMORY_SCOPE_SYSTEM);
      const unsigned v1 = __hip_atomic_load(c1, __ATOMIC_RELAXED, __HIP_MEMORY_SCOPE_SYSTEM);
      SCHED();
#pragma unroll 1
      for (int i = 0; i < nb; ++i) BAR6();
      SCHED();
      if (s0 != rank && rfl(v0) < call * 192u) wait_ge(c0, call * 192u, tmo, call);
      if (s1 != rank && rfl(v1) < call * 192u) wait_ge(c1, call * 192u, tmo, call);
    };
    auto bars = [&](int n) {
#pragma unroll 1
      for (int i = 0; i < n; ++i) BAR6();
    };
    if (rel == 0) {
      while (__builtin_amdgcn_s_memrealtime() - tk0 < LSTART) __builtin_amdgcn_s_sleep(4);
      bars(NKT + 1);
    } else if (rel == 1) {
      bars(NKT + 2);
      if (lane == 0) __hip_atomic_store(rdy, call, __ATOMIC_RELAXED, __HIP_MEMORY_SCOPE_SYSTEM);
    } else {
      bars(NKT - 3 - 4);
      gate(0, 4);
      bars(5 + FLAGK);
      if (lane == 0) __hip_atomic_store(rdy, call, __ATOMIC_RELAXED, __HIP_MEMORY_SCOPE_SYSTEM);
      bars(23 - FLAGK);
      gate(1, 4);
      STAMP(5);
      bars(36);
    }
  }

  if (rel == 1) {
    const unsigned call = sload(callp) + 1u;
    const int src = (rank + 1 + wid) & (NR - 1);
    bool ok = true;
    ok = wait_ge((const unsigned*)(myws + WS_OF + (size_t)(src * 32 + t32) * 128), call, tmo, call);
    const char* sp = P.ws[src] + WS_S + (size_t)((call & 1) * 256 + rank * 32 + t32) * TILE1;
    char* ob = P.out + ((size_t)(src * MS + mh * M1) * NS + ct * 128) * 2;
    const unsigned vl = lane * 16, vrow = (unsigned)((lane >> 3) * (NS * 2) + (lane & 7) * 16);
    if (ok) {
      u32x4 v[PDEP + 1];
#pragma unroll
      for (int i = 0; i < OPC + PDEP; ++i) {
        if (i < OPC) asm volatile("global_load_dwordx4 %0, %1, %2 sc0 sc1" : "=&v"(v[i % (PDEP + 1)]) : "v"(vl + i * 1024), "s"(sp) : "memory");
        if (i >= PDEP) {
          const int pc = i - PDEP, w = pc / (2 * MF1), r = pc % (2 * MF1);
          const int nld = (i < OPC ? i : OPC - 1) - pc, nst = pc < PDEP ? pc : PDEP;
          asm volatile("s_waitcnt vmcnt(%1)" : "+v"(v[pc % (PDEP + 1)]) : "i"(nld + nst) : "memory");
          store_sys(ob, vrow + (((w >> 1) * (M1 / 2) + r * 8) * NS + (w & 1) * 64) * 2, v[pc % (PDEP + 1)]);
        }
      }
    } else {
#pragma unroll 1
      for (int pc = 0; pc < OPC; ++pc) {
        const int w = pc / (2 * MF1), r = pc % (2 * MF1);
        store_sys(ob, vrow + (((w >> 1) * (M1 / 2) + r * 8) * NS + (w & 1) * 64) * 2,
                  (u32x4){0x7fc07fc0u, 0x7fc07fc0u, 0x7fc07fc0u, 0x7fc07fc0u});
      }
    }
    BAR();
  }
  if (wid == 6 && lane == 0) {
    __hip_atomic_fetch_add((unsigned*)(myws + WS_CALL) + bid, 1u, __ATOMIC_RELAXED, __HIP_MEMORY_SCOPE_AGENT);
  }
}

}

extern "C" void kernel_launch(int rank, int nranks, ncclComm_t comm,
                              hipStream_t stream,
                              void* const* d_in, const int* in_sizes, int n_in,
                              void* d_out, int out_size,
                              const int* in_dtypes, int out_dtype,
                              void* const* d_out_all,
                              void* const* d_ws_all, size_t ws_bytes) {
  if (nranks != NR || (unsigned)rank >= (unsigned)NR || ws_bytes < WS_NEED || n_in < 2) {
    fprintf(stderr, "unsupported configuration\n");
    return;
  }
  static bool init[NR] = {};
  if (!init[rank]) {
    HIP_CHECK(hipFuncSetAttribute((const void*)fused_gemm_a2a, hipFuncAttributeMaxDynamicSharedMemorySize, LDS_BYTES));
    init[rank] = true;
  }
  Params P;
  P.x = (const u16*)d_in[0];
  P.w = (const u16*)d_in[1];
  P.out = (char*)d_out;
  for (int p = 0; p < NR; ++p) P.ws[p] = (char*)d_ws_all[p];
  P.rank = rank;
  hipLaunchKernelGGL(fused_gemm_a2a, dim3(256), dim3(NTHREADS), LDS_BYTES, stream, P);
  HIP_CHECK(hipGetLastError());
}
```

```cpp
#include <hip/hip_runtime.h>
#include <rccl/rccl.h>
#include <cstdio>

#define HIP_CHECK(expr) do { hipError_t _e = (expr); if (_e != hipSuccess) { \
    fprintf(stderr, "HIP error %s at %s:%d\n", hipGetErrorString(_e), __FILE__, __LINE__); \
    return; } } while (0)

namespace {

typedef unsigned short u16;
typedef unsigned long long u64;
typedef unsigned u32x4 __attribute__((ext_vector_type(4)));
typedef short s16x4 __attribute__((ext_vector_type(4)));
typedef short s16x8 __attribute__((ext_vector_type(8)));
typedef __bf16 bf16x8 __attribute__((ext_vector_type(8)));
typedef float f32x4 __attribute__((ext_vector_type(4)));
#define LAS __attribute__((address_space(3)))

constexpr int NR = 8;
constexpr int MS = 512, KD = 4096, ND = 16384, NS = 2048;
constexpr int MF1 = 5, MF2 = 4;
constexpr int M1 = 32 * MF1, M2 = 32 * MF2;
constexpr int M1ROWS = 2 * M1, XROWS = MS - M1ROWS;
constexpr int NKT = KD / 64;
constexpr int NDMA = 2;
constexpr int NWAVES = 8;
constexpr int NTHREADS = NWAVES * 64;
constexpr int PA1 = M1 / 8 / NDMA, PA2 = M2 / 8 / NDMA, PB = 16 / NDMA;
constexpr int A_OFF = 65536, ASL = M1 * 128, E_OFF = A_OFF + 4 * ASL;
constexpr int LDS_BYTES = E_OFF + 16384;
static_assert(LDS_BYTES <= 163840, "lds");
constexpr int XBLK = XROWS * KD * 2;
constexpr int OPC = M1 / 4;
constexpr int TILE1 = OPC * 1024;
static_assert(XROWS == 6 * 32 && NR * XROWS == 12 * M2, "groups");

constexpr size_t WS_CALL = 0;
constexpr size_t WS_TMO = 4096;
constexpr size_t WS_XC = 8192;
constexpr size_t WS_OF = 16384;
constexpr size_t WS_STAMP = 65536;
constexpr size_t WS_G = 1u << 20;
constexpr size_t WS_S = WS_G + (size_t)2 * NR * XBLK;
constexpr size_t WS_NEED = WS_S + (size_t)2 * 256 * TILE1;

constexpr u64 DEADLINE = 100000000ull;
#define XTHR 2
#define PDEP 1
#define FLAGK 0

#define PACE_COOL 54
#define PACE_HOT 59
#define HOTCLK 178
#define PACE0 50

struct Params {
  const u16* x; const u16* w;
  char* out; char* ws[NR];
  int rank;
};

#define BAR() do { asm volatile("" ::: "memory"); __builtin_amdgcn_s_barrier(); asm volatile("" ::: "memory"); } while (0)
#define PACE() do { if (++pcount <= NKT + 1) { while (__builtin_amdgcn_s_memrealtime() - tk0 < (u64)(PACE0 + pcount * pace)) __builtin_amdgcn_s_sleep(1); } } while (0)
#define BAR6() do { PACE(); BAR(); } while (0)
#define SCHED() __builtin_amdgcn_sched_barrier(0)
#define WAIT_LGKM(n) __builtin_amdgcn_s_waitcnt(0xC07F | ((n) << 8))
#define MFMA_DRAIN() asm volatile("s_nop 7\n\ts_nop 7\n\ts_nop 3" ::: "memory")
template <int N> __device__ __forceinline__ void waitv() { static_assert(N <= 63, "vmcnt"); asm volatile("s_waitcnt vmcnt(%0)" :: "i"(N) : "memory"); }

template <int OFF, int SYS> __device__ __forceinline__ void glds(const char* sbase, unsigned voff, unsigned ldsbase) {
#define WFL " nt"
  if (SYS == 2) asm volatile("s_add_u32 m0, %2, %3\n\ts_nop 0\n\tglobal_load_lds_dwordx4 %0, %1" WFL :: "v"(voff), "s"(sbase), "s"(ldsbase), "i"(OFF) : "memory", "scc");
  else if (SYS == 1) asm volatile("s_add_u32 m0, %2, %3\n\ts_nop 0\n\tglobal_load_lds_dwordx4 %0, %1 sc0 sc1" :: "v"(voff), "s"(sbase), "s"(ldsbase), "i"(OFF) : "memory", "scc");
  else asm volatile("s_add_u32 m0, %2, %3\n\ts_nop 0\n\tglobal_load_lds_dwordx4 %0, %1" :: "v"(voff), "s"(sbase), "s"(ldsbase), "i"(OFF) : "memory", "scc");
}
template <int I, int N, int BASE, int SYS> struct Pieces {
  static __device__ __forceinline__ void go(const char* sbase, const unsigned* voff, unsigned ldsbase) {
    glds<BASE + I * 1024 * NDMA, SYS>(sbase, voff[I], ldsbase);
    Pieces<I + 1, N, BASE, SYS>::go(sbase, voff, ldsbase);
  }
};
template <int N, int BASE, int SYS> struct Pieces<N, N, BASE, SYS> { static __device__ __forceinline__ void go(const char*, const unsigned*, unsigned) {} };

__device__ __forceinline__ void store_sys(char* sbase, unsigned voff, u32x4 v) {
  asm volatile("global_store_dwordx4 %0, %1, %2 sc0 sc1\n\ts_nop 1" :: "v"(voff), "v"(v), "s"(sbase) : "memory");
}
__device__ __forceinline__ unsigned cvt_pk_bf16(float lo, float hi) {
  unsigned r; asm("v_cvt_pk_bf16_f32 %0, %1, %2" : "=v"(r) : "v"(lo), "v"(hi)); return r;
}
__device__ __forceinline__ float relu_nan(float v) { return __builtin_elementwise_maximum(v, 0.f); }
__device__ __forceinline__ unsigned rfl(unsigned v) { return (unsigned)__builtin_amdgcn_readfirstlane(v); }
__device__ __forceinline__ unsigned sload(const char* p) {
  unsigned r; asm volatile("s_load_dword %0, %1, 0x0\n\ts_waitcnt lgkmcnt(0)" : "=s"(r) : "s"(p) : "memory"); return r;
}

__device__ __forceinline__ bool wait_ge(const unsigned* flag, unsigned want, unsigned* tmo, unsigned call) {
  const u64 t0 = __builtin_amdgcn_s_memrealtime();
  unsigned polls = 0;
  while (rfl(__hip_atomic_load(flag, __ATOMIC_RELAXED, __HIP_MEMORY_SCOPE_SYSTEM)) < want) {
    if ((++polls & 15u) == 0u &&
        (rfl(__hip_atomic_load(tmo, __ATOMIC_RELAXED, __HIP_MEMORY_SCOPE_AGENT)) == call ||
         __builtin_amdgcn_s_memrealtime() - t0 > DEADLINE)) {
      __hip_atomic_store(tmo, call, __ATOMIC_RELAXED, __HIP_MEMORY_SCOPE_AGENT);
      waitv<0>();
      return false;
    }
    __builtin_amdgcn_s_sleep(2);
  }
  return true;
}

template <int MF> struct Frag { bf16x8 a[MF]; s16x4 lo[4], hi[4]; };

template <int MF>
__device__ __forceinline__ void read_one(Frag<MF>& f, int i, LAS char* lds, int s, int h, const unsigned (&ra)[2], const unsigned (&rb)[4]) {
  if (i >= 2 && i < 2 + MF) {
    f.a[i - 2] = *(LAS bf16x8*)(lds + ra[h] + s * ASL + (i - 2) * 2048);
  } else {
    const int k = i < 2 ? i : i - MF, bn = k >> 1;
    s16x4 v = __builtin_amdgcn_ds_read_tr16_b64_v4i16((LAS s16x4*)(lds + rb[bn] + s * 16384 + h * 8192 + (k & 1) * 1024));
    if (k & 1) f.hi[bn] = v; else f.lo[bn] = v;
  }
}
template <int MF, bool RD>
__device__ __forceinline__ void half_step(f32x4 (&acc)[MF][4], const Frag<MF>& f, Frag<MF>& g, LAS char* lds, int s, int h,
                                          const unsigned (&ra)[2], const unsigned (&rb)[4]) {
#pragma unroll
  for (int i = 0; i < 4 * MF; ++i) {
    const int bn = i / MF, mi = (bn & 1) ? MF - 1 - i % MF : i % MF;
    const s16x8 bv = __builtin_shufflevector(f.lo[bn], f.hi[bn], 0, 1, 2, 3, 4, 5, 6, 7);
    asm volatile("v_mfma_f32_16x16x32_bf16 %0, %1, %2, %0" : "+v"(acc[mi][bn]) : "v"(bv), "v"(f.a[mi]));
    if (RD && i < 8 + MF) read_one<MF>(g, i, lds, s, h, ra, rb);
    SCHED();
  }
}
template <int MF, int S, bool NEXT>
__device__ __forceinline__ void ktile(f32x4 (&acc)[MF][4], Frag<MF>& f0, Frag<MF>& f1, LAS char* lds,
                                      const unsigned (&ra)[2], const unsigned (&rb)[4]) {
  WAIT_LGKM(0);
  half_step<MF, true>(acc, f0, f1, lds, S, 1, ra, rb);
  WAIT_LGKM(0);
  if (S == FLAGK) waitv<0>();
  BAR();
  SCHED();
  half_step<MF, NEXT>(acc, f1, f0, lds, (S + 1) & 3, 0, ra, rb);
}
__device__ __forceinline__ u32x4 pack8(const f32x4& v0, const f32x4& v1, bool odd) {
  unsigned a0 = cvt_pk_bf16(relu_nan(v0[0]), relu_nan(v0[1])), a1 = cvt_pk_bf16(relu_nan(v0[2]), relu_nan(v0[3]));
  unsigned b0 = cvt_pk_bf16(relu_nan(v1[0]), relu_nan(v1[1])), b1 = cvt_pk_bf16(relu_nan(v1[2]), relu_nan(v1[3]));
  u32x4 o;
  o[0] = odd ? b0 : a0; o[1] = odd ? b1 : a1; o[2] = odd ? a0 : b0; o[3] = odd ? a1 : b1;
  return o;
}
template <int MF>
__device__ __forceinline__ void run_tile(LAS char* lds, int wid, int lane, char* ob, unsigned pitch, const unsigned* tmo, unsigned call) {
  const int wr = wid >> 1, wc = wid & 1, fr = lane & 15, fq = lane >> 4, q = fr >> 2, p = lane & 3;
  const unsigned rb00 = (8 * fq + q) * 256 + ((((2 * wc) ^ q) << 2) | (p ^ (fq & 1))) * 16 + (p & 1) * 8;
  const unsigned rb[4] = {rb00, rb00 ^ 8u, rb00 ^ 64u, rb00 ^ 72u};
  const unsigned rab = A_OFF + (wr * (MF * 16) + fr) * 128 + (unsigned)((fq ^ (fr >> 1)) * 16);
  const unsigned ra[2] = {rab, rab ^ 64u};
  Frag<MF> f0, f1;
#pragma unroll
  for (int i = 0; i < 8 + MF; ++i) read_one<MF>(f0, i, lds, 0, 0, ra, rb);
  f32x4 acc[MF][4];
#pragma unroll
  for (int i = 0; i < MF; ++i)
#pragma unroll
    for (int b = 0; b < 4; ++b) { acc[i][b] = (f32x4){0.f, 0.f, 0.f, 0.f}; asm volatile("" : "+v"(acc[i][b])); }
  asm volatile("s_nop 1" ::: "memory");
  SCHED();
  for (int j = 0; j < NKT - 4; j += 4) {
    ktile<MF, 0, true>(acc, f0, f1, lds, ra, rb);
    ktile<MF, 1, true>(acc, f0, f1, lds, ra, rb);
    ktile<MF, 2, true>(acc, f0, f1, lds, ra, rb);
    ktile<MF, 3, true>(acc, f0, f1, lds, ra, rb);
  }
  unsigned tmo_v = call + 1u;
  if (MF == MF2) tmo_v = __hip_atomic_load(tmo, __ATOMIC_RELAXED, __HIP_MEMORY_SCOPE_AGENT);
  SCHED();
  ktile<MF, 0, true>(acc, f0, f1, lds, ra, rb);
  ktile<MF, 1, true>(acc, f0, f1, lds, ra, rb);
  ktile<MF, 2, true>(acc, f0, f1, lds, ra, rb);
  ktile<MF, 3, false>(acc, f0, f1, lds, ra, rb);
  MFMA_DRAIN();
  SCHED();
  const bool odd = fq & 1, bad = tmo_v == call;
  LAS char* sc = lds + E_OFF + wid * 4096;
  const unsigned wo = fr * 128 + ((fq ^ (fr & 7)) * 16), ro = (lane >> 3) * 128 + (((lane & 7) ^ (lane >> 3)) * 16);
  unsigned vo = (lane >> 3) * pitch + (lane & 7) * 16;
#pragma unroll
  for (int m0 = 0; m0 < MF; m0 += 2) {
#pragma unroll
    for (int mi = m0; mi < m0 + 2 && mi < MF; ++mi) {
      *(LAS u32x4*)(sc + (mi - m0) * 2048 + wo) = pack8(acc[mi][0], acc[mi][1], odd);
      *(LAS u32x4*)(sc + (mi - m0) * 2048 + (wo ^ 64u)) = pack8(acc[mi][2], acc[mi][3], odd);
    }
    asm volatile("" ::: "memory");
#pragma unroll
    for (int r = 0; r < (m0 + 2 <= MF ? 4 : 2); ++r) {
      u32x4 o = *(LAS u32x4*)(sc + r * 1024 + ro);
      if (MF == MF2 && bad) o = (u32x4){0x7fc07fc0u, 0x7fc07fc0u, 0x7fc07fc0u, 0x7fc07fc0u};
      store_sys(ob, vo, o);
      vo += 8 * pitch;
    }
    asm volatile("" ::: "memory");
  }
}

template <int D>
__device__ __forceinline__ void pull_tile(const char* sp, char* ob, unsigned vl, unsigned vrow) {
  u32x4 v[D + 1];
#pragma unroll
  for (int i = 0; i < OPC + D; ++i) {
    if (i < OPC) asm volatile("global_load_dwordx4 %0, %1, %2 sc0 sc1" : "=&v"(v[i % (D + 1)]) : "v"(vl + i * 1024), "s"(sp) : "memory");
    if (i >= D) {
      const int pc = i - D, w = pc / (2 * MF1), r = pc % (2 * MF1);
      const int nld = (i < OPC ? i : OPC - 1) - pc, nst = pc < D ? pc : D;
      asm volatile("s_waitcnt vmcnt(%1)" : "+v"(v[pc % (D + 1)]) : "i"(nld + nst) : "memory");
      store_sys(ob, vrow + (((w >> 1) * (M1 / 2) + r * 8) * NS + (w & 1) * 64) * 2, v[pc % (D + 1)]);
    }
  }
}

#define STAMP(i) do {} while (0)
#define STAMPC(i) do {} while (0)

__global__ void __launch_bounds__(NTHREADS, 2) fused_gemm_a2a(const Params P) {
  extern __shared__ __attribute__((aligned(16))) char smem_[];
  LAS char* lds = (LAS char*)smem_;
  const int tid = threadIdx.x, lane = tid & 63, wid = __builtin_amdgcn_readfirstlane(tid >> 6);
  const int bid = blockIdx.x, xcd = bid & 7, idx = bid >> 3, rank = P.rank;
  const int dst1 = idx >> 2, t32 = bid & 31;
  const int rel = (dst1 - rank) & (NR - 1);
#define CMIX 1
  auto rcol = [&](int owner) { const int hb = (t32 >> 4) & 1; return hb * 8 + ((xcd + CMIX * (owner * 2 + hb)) & 7); };
  const int mh = rel == 0 ? (xcd & 1) : (idx & 1), ct = rel == 0 ? (xcd >> 1) * 4 + (idx & 3) : rcol(dst1);
  const bool hasG = rel >= 2;
  const int ci = (rel < 2 ? 0 : rel - 2) * 4 + (idx & 3), m6 = ci % 6, n4 = ci / 6;
  const int mt2 = (xcd & 1) * 6 + m6, nl2 = (xcd >> 1) * 4 + n4;
  char* myws = P.ws[rank];
  unsigned* tmo = (unsigned*)(myws + WS_TMO);
  const char* callp = myws + WS_CALL + bid * 4;
  if (wid == 0) { STAMP(0); STAMPC(8); }
  const u64 tk0 = __builtin_amdgcn_s_memrealtime();

  if (rel == 0 && (wid < 4 || wid >= 6)) {
    static_assert(XTHR < NR - 1, "xthr");
    const int xw = wid < 4 ? wid : wid - 2, row = t32 * 6 + xw;
    const unsigned vl = lane * 16;
    u32x4 xv[8];
    const char* xs = (const char*)P.x + (size_t)(M1ROWS + row) * KD * 2 + vl;
#pragma unroll
    for (int i = 0; i < 8; ++i) xv[i] = *(const u32x4*)(xs + i * 1024);
    const unsigned call = sload(callp) + 1u;
    const size_t goff = WS_G + (size_t)((call & 1) * NR + rank) * XBLK + (size_t)row * KD * 2;
    char* g[NR - 1];
#pragma unroll
    for (int j = 0; j < NR - 1; ++j) { int pj = j + xw; if (pj >= NR - 1) pj -= NR - 1; g[j] = P.ws[(rank + 1 + pj) & (NR - 1)] + goff; }
    char* sigp;
    {
      const int peer = (rank + 1 + lane) & (NR - 1);
      char* pw = P.ws[0];
#pragma unroll
      for (int r = 1; r < NR; ++r) pw = (peer == r) ? P.ws[r] : pw;
      sigp = pw + WS_XC + rank * 128;
    }
#pragma unroll
    for (int i = 0; i < 8; ++i) asm volatile("" : "+v"(xv[i]));
#pragma unroll
    for (int i = 0; i < 8; ++i) {
      unsigned vo = vl + i * 1024;
      asm volatile("" : "+v"(vo));
#pragma unroll
      for (int j = 0; j < NR - 1; ++j) { store_sys(g[j], vo, xv[i]); waitv<XTHR>(); }
      if (i == 4 && lane < NR - 1) __hip_atomic_fetch_add((unsigned*)sigp, 1u, __ATOMIC_RELAXED, __HIP_MEMORY_SCOPE_SYSTEM);
    }
    waitv<0>();
    if (lane < NR - 1) __hip_atomic_fetch_add((unsigned*)(sigp + NR * 128), 1u, __ATOMIC_RELAXED, __HIP_MEMORY_SCOPE_SYSTEM);
    if (wid == 0) STAMP(6);
  }
  if (wid == 7) return;

  if (wid < 4) {
    const int wr = wid >> 1, wc = wid & 1;
    const unsigned call = sload(callp) + 1u;
    BAR();
    if (wid == 0) STAMP(1);
    WAIT_LGKM(0);
    {
      char* ob = rel != 0 ? myws + WS_S + (size_t)((call & 1) * 256 + bid) * TILE1 + wid * (MF1 * 2048)
                          : P.out + ((size_t)(rank * MS + mh * M1 + wr * (M1 / 2)) * NS + ct * 128 + wc * 64) * 2;
      run_tile<MF1>(lds, wid, lane, ob, rel != 0 ? 128u : (unsigned)(NS * 2), tmo, call);
      if (wid == 0) { STAMP(rel == 0 ? 3 : 2); WAIT_LGKM(0); }
    }
    if (hasG) {
      const int blk = mt2 * 2 + wr;
      char* ob = P.out + ((size_t)((blk / 3) * MS + M1ROWS + (blk % 3) * 64) * NS + nl2 * 128 + wc * 64) * 2;
      run_tile<MF2>(lds, wid, lane, ob, NS * 2, tmo, call);
      if (wid == 0) STAMP(3);
    }
    if (rel == 1) {
      waitv<0>();
      BAR();
    }
  } else if (wid < 4 + NDMA) {
    const int dw = wid - 4;
    unsigned va[PA1], vb[PB];
#pragma unroll
    for (int i = 0; i < PA1; ++i) {
      const int pc = i * NDMA + dw;
      va[i] = (unsigned)((pc * 8 + (lane >> 3)) * (KD * 2) + (((lane & 7) ^ ((4 * pc + (lane >> 4)) & 7)) * 16));
    }
#pragma unroll
    for (int i = 0; i < PB; ++i) {
      const int pc = i * NDMA + dw;
      vb[i] = (unsigned)((pc * 4 + (lane >> 4)) * (ND * 2) + (((lane & 15) ^ (((lane >> 4) << 2) | ((pc >> 1) & 1))) * 16));
    }
    const unsigned ldsb = (unsigned)(size_t)lds + dw * 1024;
    const char* a1 = (const char*)P.x + (size_t)(mh * M1) * KD * 2;
#define KX1 (mh | (((idx >> 1) & 3) << 1))
#define KX2 ((m6 ^ n4) | ((xcd & 3) << 3))
#define LROT 0
    const int kx1 = rel == 0 ? ((idx & 3) | ((xcd & 3) << 3)) : (KX1), kr1 = rel == 0 ? LROT : 0, kx2 = KX2;
#define LD_B(wb, kt, S) Pieces<0, PB, (S) * 16384, false>::go((wb) + (size_t)(kt) * 64 * ND * 2, vb, ldsb)
#define KT1(j) ((((j) + kr1) & (NKT - 1)) ^ kx1)
#define LD_B1 LD_B
#define LD_1(j, S) do { const int kt_ = KT1(j); LD_B1(w1, kt_, S); Pieces<0, PA1, A_OFF + (S) * ASL, false>::go(a1 + kt_ * 128, va, ldsb); } while (0)
#define LD_A2(j, S) do { Pieces<0, PA2 / 2, A_OFF + (S) * ASL, true>::go(a2lo + ((j) ^ kx2) * 128, va, ldsb); \
                         Pieces<PA2 / 2, PA2, A_OFF + (S) * ASL, true>::go(a2hi + ((j) ^ kx2) * 128, va, ldsb); } while (0)
#define QUAD2(j, W0, W1, W2, W3) do { \
      waitv<W0>(); BAR(); LD_B(w2, (j) ^ kx2, 0); LD_A2(j, 0); \
      waitv<W1>(); BAR(); LD_B(w2, ((j) + 1) ^ kx2, 1); LD_A2((j) + 1, 1); \
      waitv<W2>(); BAR(); LD_B(w2, ((j) + 2) ^ kx2, 2); LD_A2((j) + 2, 2); \
      waitv<W3>(); BAR(); LD_B(w2, ((j) + 3) ^ kx2, 3); LD_A2((j) + 3, 3); } while (0)
    constexpr int P1 = PA1 + PB, P2 = PA2 + PB;
    {
      const char* w1 = (const char*)P.w + (size_t)(dst1 * 16 + ct) * 256;
      LD_1(0, 0); LD_1(1, 1); LD_1(2, 2);
      waitv<2 * P1>();
      if (rel == 0) {
        LD_1(3, 3);
        while (__builtin_amdgcn_s_memrealtime() - tk0 < 370) __builtin_amdgcn_s_sleep(2);
        const u64 c0 = __builtin_amdgcn_s_memtime(), r0 = __builtin_amdgcn_s_memrealtime();
        while (__builtin_amdgcn_s_memrealtime() - tk0 < 780) __builtin_amdgcn_s_sleep(2);
        const u64 c1 = __builtin_amdgcn_s_memtime(), r1 = __builtin_amdgcn_s_memrealtime();
        const int pace = ((unsigned)(c1 - c0) * 10u > (unsigned)(r1 - r0) * (unsigned)HOTCLK) ? PACE_HOT : (PACE_COOL > 55 ? PACE_COOL : 55);
        while (__builtin_amdgcn_s_memrealtime() - tk0 < (u64)(NKT * pace + 130)) __builtin_amdgcn_s_sleep(4);
        BAR();
      } else {
        BAR();
        LD_1(3, 3);
      }
#pragma unroll 1
      for (int j = 4; j < NKT; j += 4) {
        waitv<2 * P1>(); BAR(); LD_1(j, 0);
        waitv<2 * P1>(); BAR(); LD_1(j + 1, 1);
        waitv<2 * P1>(); BAR(); LD_1(j + 2, 2);
        waitv<2 * P1>(); BAR(); LD_1(j + 3, 3);
      }
    }
    if (hasG) {
      const unsigned call = sload(callp) + 1u;
      const char* w2 = (const char*)P.w + (size_t)(rank * 16 + nl2) * 256;
      const int b0 = mt2 * 2, b1 = b0 + 1;
      const char* gb = myws + WS_G + (size_t)((call & 1) * NR) * XBLK;
      const char* xb = (const char*)P.x + (size_t)M1ROWS * KD * 2;
      const char* a2lo = (b0 / 3 == rank) ? xb + (size_t)((b0 % 3) * 64) * KD * 2 : gb + (size_t)(b0 * 64) * KD * 2;
      const char* a2hi = ((b1 / 3 == rank) ? xb + (size_t)((b1 % 3) * 64) * KD * 2 : gb + (size_t)(b1 * 64) * KD * 2) - (size_t)64 * KD * 2;
      QUAD2(0, 2 * P1, P1 + P2, 2 * P2, 2 * P2);
#pragma unroll 1
      for (int j = 4; j < NKT; j += 4) QUAD2(j, 2 * P2, 2 * P2, 2 * P2, 2 * P2);
      waitv<2 * P2>(); BAR();
      waitv<P2>(); BAR();
    } else {
      waitv<2 * P1>(); BAR();
      waitv<P1>(); BAR();
    }
    waitv<0>(); BAR();
    BAR();
    if (rel == 1) BAR();
  } else {
    int tcount = -1; (void)tcount;
    int pcount = rel == 0 ? 1000 : 0, pace = PACE_COOL;
    const unsigned call = sload(callp) + 1u;
    unsigned* rdy = (unsigned*)(P.ws[dst1] + WS_OF + (size_t)(rank * 32 + t32) * 128);
    const int s0 = (mt2 * 2) / 3, s1 = (mt2 * 2 + 1) / 3;
    auto gate = [&](int h, int nb) {
      const unsigned* c0 = (const unsigned*)(myws + WS_XC + (h * NR + s0) * 128);
      const unsigned* c1 = (const unsigned*)(myws + WS_XC + (h * NR + s1) * 128);
      const unsigned v0 = __hip_atomic_load(c0, __ATOMIC_RELAXED, __HIP_MEMORY_SCOPE_SYSTEM);
      const unsigned v1 = __hip_atomic_load(c1, __ATOMIC_RELAXED, __HIP_MEMORY_SCOPE_SYSTEM);
      SCHED();
#pragma unroll 1
      for (int i = 0; i < nb; ++i) BAR6();
      SCHED();
      if (s0 != rank && rfl(v0) < call * 192u) wait_ge(c0, call * 192u, tmo, call);
      if (s1 != rank && rfl(v1) < call * 192u) wait_ge(c1, call * 192u, tmo, call);
    };
    auto bars = [&](int n) {
#pragma unroll 1
      for (int i = 0; i < n; ++i) BAR6();
    };
    if (rel != 0) {
      bars(3);
      const u64 c0 = __builtin_amdgcn_s_memtime(), r0 = __builtin_amdgcn_s_memrealtime();
      bars(8);
      const u64 c1 = __builtin_amdgcn_s_memtime(), r1 = __builtin_amdgcn_s_memrealtime();
      if ((unsigned)(c1 - c0) * 10u > (unsigned)(r1 - r0) * (unsigned)HOTCLK) pace = PACE_HOT;
    }
    if (rel == 0) {
      bars(NKT + 1);
    } else if (rel == 1) {
      bars(NKT + 2 - 11);
      if (lane == 0) __hip_atomic_store(rdy, call, __ATOMIC_RELAXED, __HIP_MEMORY_SCOPE_SYSTEM);
    } else {
      bars(NKT - 3 - 4 - 11);
      gate(0, 4);
      bars(5 + FLAGK);
      if (lane == 0) __hip_atomic_store(rdy, call, __ATOMIC_RELAXED, __HIP_MEMORY_SCOPE_SYSTEM);
      bars(23 - FLAGK);
      gate(1, 4);
      STAMP(5);
      bars(36);
    }
  }

  if (rel == 1) {
    const unsigned call = sload(callp) + 1u;
    const int src = (rank + 1 + wid) & (NR - 1);
    bool ok = true;
    ok = wait_ge((const unsigned*)(myws + WS_OF + (size_t)(src * 32 + t32) * 128), call, tmo, call);
    if (wid == 3) STAMP(10);
    const char* sp = P.ws[src] + WS_S + (size_t)((call & 1) * 256 + rank * 32 + t32) * TILE1;
    char* ob = P.out + ((size_t)(src * MS + mh * M1) * NS + rcol(rank) * 128) * 2;
    const unsigned vl = lane * 16, vrow = (unsigned)((lane >> 3) * (NS * 2) + (lane & 7) * 16);
    if (ok) {
      pull_tile<PDEP>(sp, ob, vl, vrow);
    } else {
#pragma unroll 1
      for (int pc = 0; pc < OPC; ++pc) {
        const int w = pc / (2 * MF1), r = pc % (2 * MF1);
        store_sys(ob, vrow + (((w >> 1) * (M1 / 2) + r * 8) * NS + (w & 1) * 64) * 2,
                  (u32x4){0x7fc07fc0u, 0x7fc07fc0u, 0x7fc07fc0u, 0x7fc07fc0u});
      }
    }
    BAR();
  }
  if (wid == 6 && lane == 0) {
    __hip_atomic_fetch_add((unsigned*)(myws + WS_CALL) + bid, 1u, __ATOMIC_RELAXED, __HIP_MEMORY_SCOPE_AGENT);
  }
}

}

extern "C" void kernel_launch(int rank, int nranks, ncclComm_t comm,
                              hipStream_t stream,
                              void* const* d_in, const int* in_sizes, int n_in,
                              void* d_out, int out_size,
                              const int* in_dtypes, int out_dtype,
                              void* const* d_out_all,
                              void* const* d_ws_all, size_t ws_bytes) {
  if (nranks != NR || (unsigned)rank >= (unsigned)NR || ws_bytes < WS_NEED || n_in < 2) {
    fprintf(stderr, "unsupported configuration\n");
    return;
  }
  static bool init[NR] = {};
  if (!init[rank]) {
    HIP_CHECK(hipFuncSetAttribute((const void*)fused_gemm_a2a, hipFuncAttributeMaxDynamicSharedMemorySize, LDS_BYTES));
    init[rank] = true;
  }
  Params P;
  P.x = (const u16*)d_in[0];
  P.w = (const u16*)d_in[1];
  P.out = (char*)d_out;
  for (int p = 0; p < NR; ++p) P.ws[p] = (char*)d_ws_all[p];
  P.rank = rank;
  hipLaunchKernelGGL(fused_gemm_a2a, dim3(256), dim3(NTHREADS), LDS_BYTES, stream, P);
  HIP_CHECK(hipGetLastError());
}
```

```cpp
#include <hip/hip_runtime.h>
#include <rccl/rccl.h>
#include <cstdio>

#define HIP_CHECK(expr) do { hipError_t _e = (expr); if (_e != hipSuccess) { \
    fprintf(stderr, "HIP error %s at %s:%d\n", hipGetErrorString(_e), __FILE__, __LINE__); \
    return; } } while (0)

namespace {

typedef unsigned short u16;
typedef unsigned long long u64;
typedef unsigned u32x4 __attribute__((ext_vector_type(4)));
typedef short s16x4 __attribute__((ext_vector_type(4)));
typedef short s16x8 __attribute__((ext_vector_type(8)));
typedef __bf16 bf16x8 __attribute__((ext_vector_type(8)));
typedef float f32x4 __attribute__((ext_vector_type(4)));
#define LAS __attribute__((address_space(3)))

constexpr int NR = 8;
constexpr int MS = 512, KD = 4096, ND = 16384, NS = 2048;
constexpr int MF1 = 5, MF2 = 4;
constexpr int M1 = 32 * MF1, M2 = 32 * MF2;
constexpr int M1ROWS = 2 * M1, XROWS = MS - M1ROWS;
constexpr int NKT = KD / 64;
constexpr int NDMA = 2;
constexpr int NWAVES = 8;
constexpr int NTHREADS = NWAVES * 64;
constexpr int PA1 = M1 / 8 / NDMA, PA2 = M2 / 8 / NDMA, PB = 16 / NDMA;
constexpr int A_OFF = 65536, ASL = M1 * 128, E_OFF = A_OFF + 4 * ASL;
constexpr int LDS_BYTES = E_OFF + 16384;
static_assert(LDS_BYTES <= 163840, "lds");
constexpr int XBLK = XROWS * KD * 2;
constexpr int OPC = M1 / 4;
constexpr int TILE1 = OPC * 1024;
static_assert(XROWS == 6 * 32 && NR * XROWS == 12 * M2, "groups");

constexpr size_t WS_CALL = 0;
constexpr size_t WS_TMO = 4096;
constexpr size_t WS_XC = 8192;
constexpr size_t WS_OF = 16384;
constexpr size_t WS_G = 1u << 20;
constexpr size_t WS_S = WS_G + (size_t)2 * NR * XBLK;
constexpr size_t WS_NEED = WS_S + (size_t)2 * 256 * TILE1;

constexpr u64 DEADLINE = 100000000ull;
#define XTHR 2
#define PDEP 1
#define FLAGK 0

#define PACE_COOL 54
#define PACE_HOT 59
#define CLK_COOL 176
#define CLK_HOT 186
#define CLKW 12
#define LOFF 90
#define PACE0 50

struct Params {
  const u16* x; const u16* w;
  char* out; char* ws[NR];
  int rank;
};

#define BAR() do { asm volatile("" ::: "memory"); __builtin_amdgcn_s_barrier(); asm volatile("" ::: "memory"); } while (0)
#define PACE() do { if (++pcount <= NKT + 1) { while (__builtin_amdgcn_s_memrealtime() - tk0 < (u64)(PACE0 + pcount * pace)) __builtin_amdgcn_s_sleep(1); } } while (0)
#define BAR6() do { PACE(); BAR(); } while (0)
#define SCHED() __builtin_amdgcn_sched_barrier(0)
#define WAIT_LGKM(n) __builtin_amdgcn_s_waitcnt(0xC07F | ((n) << 8))
#define MFMA_DRAIN() asm volatile("s_nop 7\n\ts_nop 7\n\ts_nop 3" ::: "memory")
template <int N> __device__ __forceinline__ void waitv() { static_assert(N <= 63, "vmcnt"); asm volatile("s_waitcnt vmcnt(%0)" :: "i"(N) : "memory"); }

template <int OFF, bool SYS> __device__ __forceinline__ void glds(const char* sbase, unsigned voff, unsigned ldsbase) {
  if (SYS) asm volatile("s_add_u32 m0, %2, %3\n\ts_nop 0\n\tglobal_load_lds_dwordx4 %0, %1 sc0 sc1" :: "v"(voff), "s"(sbase), "s"(ldsbase), "i"(OFF) : "memory", "scc");
  else asm volatile("s_add_u32 m0, %2, %3\n\ts_nop 0\n\tglobal_load_lds_dwordx4 %0, %1" :: "v"(voff), "s"(sbase), "s"(ldsbase), "i"(OFF) : "memory", "scc");
}
template <int I, int N, int BASE, bool SYS> struct Pieces {
  static __device__ __forceinline__ void go(const char* sbase, const unsigned* voff, unsigned ldsbase) {
    glds<BASE + I * 1024 * NDMA, SYS>(sbase, voff[I], ldsbase);
    Pieces<I + 1, N, BASE, SYS>::go(sbase, voff, ldsbase);
  }
};
template <int N, int BASE, bool SYS> struct Pieces<N, N, BASE, SYS> { static __device__ __forceinline__ void go(const char*, const unsigned*, unsigned) {} };

__device__ __forceinline__ void store_sys(char* sbase, unsigned voff, u32x4 v) {
  asm volatile("global_store_dwordx4 %0, %1, %2 sc0 sc1\n\ts_nop 1" :: "v"(voff), "v"(v), "s"(sbase) : "memory");
}
__device__ __forceinline__ unsigned cvt_pk_bf16(float lo, float hi) {
  unsigned r; asm("v_cvt_pk_bf16_f32 %0, %1, %2" : "=v"(r) : "v"(lo), "v"(hi)); return r;
}
__device__ __forceinline__ float relu_nan(float v) { return __builtin_elementwise_maximum(v, 0.f); }
__device__ __forceinline__ unsigned rfl(unsigned v) { return (unsigned)__builtin_amdgcn_readfirstlane(v); }
__device__ __forceinline__ unsigned sload(const char* p) {
  unsigned r; asm volatile("s_load_dword %0, %1, 0x0\n\ts_waitcnt lgkmcnt(0)" : "=s"(r) : "s"(p) : "memory"); return r;
}

__device__ __forceinline__ int pace_of(u64 cycles, u64 ticks) {
  const int f = (int)((unsigned)cycles * 10u / ((unsigned)ticks | 1u));
  const int p = PACE_COOL + (f - CLK_COOL) * (PACE_HOT - PACE_COOL) / (CLK_HOT - CLK_COOL);
  return p < PACE_COOL ? PACE_COOL : p > PACE_HOT ? PACE_HOT : p;
}

__device__ __forceinline__ bool wait_ge(const unsigned* flag, unsigned want, unsigned* tmo, unsigned call) {
  const u64 t0 = __builtin_amdgcn_s_memrealtime();
  unsigned polls = 0;
  while (rfl(__hip_atomic_load(flag, __ATOMIC_RELAXED, __HIP_MEMORY_SCOPE_SYSTEM)) < want) {
    if ((++polls & 15u) == 0u &&
        (rfl(__hip_atomic_load(tmo, __ATOMIC_RELAXED, __HIP_MEMORY_SCOPE_AGENT)) == call ||
         __builtin_amdgcn_s_memrealtime() - t0 > DEADLINE)) {
      __hip_atomic_store(tmo, call, __ATOMIC_RELAXED, __HIP_MEMORY_SCOPE_AGENT);
      waitv<0>();
      return false;
    }
    __builtin_amdgcn_s_sleep(2);
  }
  return true;
}

template <int MF> struct Frag { bf16x8 a[MF]; s16x4 lo[4], hi[4]; };

template <int MF>
__device__ __forceinline__ void read_one(Frag<MF>& f, int i, LAS char* lds, int s, int h, const unsigned (&ra)[2], const unsigned (&rb)[4]) {
  if (i >= 2 && i < 2 + MF) {
    f.a[i - 2] = *(LAS bf16x8*)(lds + ra[h] + s * ASL + (i - 2) * 2048);
  } else {
    const int k = i < 2 ? i : i - MF, bn = k >> 1;
    s16x4 v = __builtin_amdgcn_ds_read_tr16_b64_v4i16((LAS s16x4*)(lds + rb[bn] + s * 16384 + h * 8192 + (k & 1) * 1024));
    if (k & 1) f.hi[bn] = v; else f.lo[bn] = v;
  }
}
template <int MF, bool RD>
__device__ __forceinline__ void half_step(f32x4 (&acc)[MF][4], const Frag<MF>& f, Frag<MF>& g, LAS char* lds, int s, int h,
                                          const unsigned (&ra)[2], const unsigned (&rb)[4]) {
#pragma unroll
  for (int i = 0; i < 4 * MF; ++i) {
    const int bn = i / MF, mi = (bn & 1) ? MF - 1 - i % MF : i % MF;
    const s16x8 bv = __builtin_shufflevector(f.lo[bn], f.hi[bn], 0, 1, 2, 3, 4, 5, 6, 7);
    asm volatile("v_mfma_f32_16x16x32_bf16 %0, %1, %2, %0" : "+v"(acc[mi][bn]) : "v"(bv), "v"(f.a[mi]));
    if (RD && i < 8 + MF) read_one<MF>(g, i, lds, s, h, ra, rb);
    SCHED();
  }
}
template <int MF, int S, bool NEXT>
__device__ __forceinline__ void ktile(f32x4 (&acc)[MF][4], Frag<MF>& f0, Frag<MF>& f1, LAS char* lds,
                                      const unsigned (&ra)[2], const unsigned (&rb)[4]) {
  WAIT_LGKM(0);
  half_step<MF, true>(acc, f0, f1, lds, S, 1, ra, rb);
  WAIT_LGKM(0);
  if (S == FLAGK) waitv<0>();
  BAR();
  SCHED();
  half_step<MF, NEXT>(acc, f1, f0, lds, (S + 1) & 3, 0, ra, rb);
}
__device__ __forceinline__ u32x4 pack8(const f32x4& v0, const f32x4& v1, bool odd) {
  unsigned a0 = cvt_pk_bf16(relu_nan(v0[0]), relu_nan(v0[1])), a1 = cvt_pk_bf16(relu_nan(v0[2]), relu_nan(v0[3]));
  unsigned b0 = cvt_pk_bf16(relu_nan(v1[0]), relu_nan(v1[1])), b1 = cvt_pk_bf16(relu_nan(v1[2]), relu_nan(v1[3]));
  u32x4 o;
  o[0] = odd ? b0 : a0; o[1] = odd ? b1 : a1; o[2] = odd ? a0 : b0; o[3] = odd ? a1 : b1;
  return o;
}
template <int MF>
__device__ __forceinline__ void run_tile(LAS char* lds, int wid, int lane, char* ob, unsigned pitch, const unsigned* tmo, unsigned call) {
  const int wr = wid >> 1, wc = wid & 1, fr = lane & 15, fq = lane >> 4, q = fr >> 2, p = lane & 3;
  const unsigned rb00 = (8 * fq + q) * 256 + ((((2 * wc) ^ q) << 2) | (p ^ (fq & 1))) * 16 + (p & 1) * 8;
  const unsigned rb[4] = {rb00, rb00 ^ 8u, rb00 ^ 64u, rb00 ^ 72u};
  const unsigned rab = A_OFF + (wr * (MF * 16) + fr) * 128 + (unsigned)((fq ^ (fr >> 1)) * 16);
  const unsigned ra[2] = {rab, rab ^ 64u};
  Frag<MF> f0, f1;
#pragma unroll
  for (int i = 0; i < 8 + MF; ++i) read_one<MF>(f0, i, lds, 0, 0, ra, rb);
  f32x4 acc[MF][4];
#pragma unroll
  for (int i = 0; i < MF; ++i)
#pragma unroll
    for (int b = 0; b < 4; ++b) { acc[i][b] = (f32x4){0.f, 0.f, 0.f, 0.f}; asm volatile("" : "+v"(acc[i][b])); }
  asm volatile("s_nop 1" ::: "memory");
  SCHED();
  for (int j = 0; j < NKT - 4; j += 4) {
    ktile<MF, 0, true>(acc, f0, f1, lds, ra, rb);
    ktile<MF, 1, true>(acc, f0, f1, lds, ra, rb);
    ktile<MF, 2, true>(acc, f0, f1, lds, ra, rb);
    ktile<MF, 3, true>(acc, f0, f1, lds, ra, rb);
  }
  static_assert(FLAGK == 0, "the load below stays in flight until the epilogue");
  ktile<MF, 0, true>(acc, f0, f1, lds, ra, rb);
  unsigned tmo_v = call + 1u;
  if (MF == MF2) tmo_v = __hip_atomic_load(tmo, __ATOMIC_RELAXED, __HIP_MEMORY_SCOPE_AGENT);
  SCHED();
  ktile<MF, 1, true>(acc, f0, f1, lds, ra, rb);
  ktile<MF, 2, true>(acc, f0, f1, lds, ra, rb);
  ktile<MF, 3, false>(acc, f0, f1, lds, ra, rb);
  MFMA_DRAIN();
  SCHED();
  const bool odd = fq & 1, bad = tmo_v == call;
  LAS char* sc = lds + E_OFF + wid * 4096;
  const unsigned wo = fr * 128 + ((fq ^ (fr & 7)) * 16), ro = (lane >> 3) * 128 + (((lane & 7) ^ (lane >> 3)) * 16);
  unsigned vo = (lane >> 3) * pitch + (lane & 7) * 16;
#pragma unroll
  for (int m0 = 0; m0 < MF; m0 += 2) {
#pragma unroll
    for (int mi = m0; mi < m0 + 2 && mi < MF; ++mi) {
      *(LAS u32x4*)(sc + (mi - m0) * 2048 + wo) = pack8(acc[mi][0], acc[mi][1], odd);
      *(LAS u32x4*)(sc + (mi - m0) * 2048 + (wo ^ 64u)) = pack8(acc[mi][2], acc[mi][3], odd);
    }
    asm volatile("" ::: "memory");
#pragma unroll
    for (int r = 0; r < (m0 + 2 <= MF ? 4 : 2); ++r) {
      u32x4 o = *(LAS u32x4*)(sc + r * 1024 + ro);
      if (MF == MF2 && bad) o = (u32x4){0x7fc07fc0u, 0x7fc07fc0u, 0x7fc07fc0u, 0x7fc07fc0u};
      store_sys(ob, vo, o);
      vo += 8 * pitch;
    }
    asm volatile("" ::: "memory");
  }
}

template <int D>
__device__ __forceinline__ void pull_tile(const char* sp, char* ob, unsigned vl, unsigned vrow) {
  u32x4 v[D + 1];
#pragma unroll
  for (int i = 0; i < OPC + D; ++i) {
    if (i < OPC) asm volatile("global_load_dwordx4 %0, %1, %2 sc0 sc1" : "=&v"(v[i % (D + 1)]) : "v"(vl + i * 1024), "s"(sp) : "memory");
    if (i >= D) {
      const int pc = i - D, w = pc / (2 * MF1), r = pc % (2 * MF1);
      const int nld = (i < OPC ? i : OPC - 1) - pc, nst = pc < D ? pc : D;
      asm volatile("s_waitcnt vmcnt(%1)" : "+v"(v[pc % (D + 1)]) : "i"(nld + nst) : "memory");
      store_sys(ob, vrow + (((w >> 1) * (M1 / 2) + r * 8) * NS + (w & 1) * 64) * 2, v[pc % (D + 1)]);
    }
  }
}


__global__ void __launch_bounds__(NTHREADS, 2) fused_gemm_a2a(const Params P) {
  extern __shared__ __attribute__((aligned(16))) char smem_[];
  LAS char* lds = (LAS char*)smem_;
  const int tid = threadIdx.x, lane = tid & 63, wid = __builtin_amdgcn_readfirstlane(tid >> 6);
  const int bid = blockIdx.x, xcd = bid & 7, idx = bid >> 3, rank = P.rank;
  const int dst1 = idx >> 2, t32 = bid & 31;
  const int rel = (dst1 - rank) & (NR - 1);
#define CMIX 1
  auto rcol = [&](int owner) { const int hb = (t32 >> 4) & 1; return hb * 8 + ((xcd + CMIX * (owner * 2 + hb)) & 7); };
  const int mh = rel == 0 ? (xcd & 1) : (idx & 1), ct = rel == 0 ? (xcd >> 1) * 4 + (idx & 3) : rcol(dst1);
  const bool hasG = rel >= 2;
  const int ci = (rel < 2 ? 0 : rel - 2) * 4 + (idx & 3), m6 = ci % 6, n4 = ci / 6;
  const int mt2 = (xcd & 1) * 6 + m6, nl2 = (xcd >> 1) * 4 + n4;
  char* myws = P.ws[rank];
  unsigned* tmo = (unsigned*)(myws + WS_TMO);
  const char* callp = myws + WS_CALL + bid * 4;
  const u64 tk0 = __builtin_amdgcn_s_memrealtime();

  if (rel == 0 && (wid < 4 || wid >= 6)) {
    static_assert(XTHR < NR - 1, "xthr");
    const int xw = wid < 4 ? wid : wid - 2, row = t32 * 6 + xw;
    const unsigned vl = lane * 16;
    u32x4 xv[8];
    const char* xs = (const char*)P.x + (size_t)(M1ROWS + row) * KD * 2 + vl;
#pragma unroll
    for (int i = 0; i < 8; ++i) xv[i] = *(const u32x4*)(xs + i * 1024);
    const unsigned call = sload(callp) + 1u;
    const size_t goff = WS_G + (size_t)((call & 1) * NR + rank) * XBLK + (size_t)row * KD * 2;
    char* g[NR - 1];
#pragma unroll
    for (int j = 0; j < NR - 1; ++j) { int pj = j + xw; if (pj >= NR - 1) pj -= NR - 1; g[j] = P.ws[(rank + 1 + pj) & (NR - 1)] + goff; }
    char* sigp;
    {
      const int peer = (rank + 1 + lane) & (NR - 1);
      char* pw = P.ws[0];
#pragma unroll
      for (int r = 1; r < NR; ++r) pw = (peer == r) ? P.ws[r] : pw;
      sigp = pw + WS_XC + rank * 128;
    }
#pragma unroll
    for (int i = 0; i < 8; ++i) asm volatile("" : "+v"(xv[i]));
#pragma unroll
    for (int i = 0; i < 8; ++i) {
      unsigned vo = vl + i * 1024;
      asm volatile("" : "+v"(vo));
#pragma unroll
      for (int j = 0; j < NR - 1; ++j) { store_sys(g[j], vo, xv[i]); waitv<XTHR>(); }
      if (i == 4 && lane < NR - 1) __hip_atomic_fetch_add((unsigned*)sigp, 1u, __ATOMIC_RELAXED, __HIP_MEMORY_SCOPE_SYSTEM);
    }
    waitv<0>();
    if (lane < NR - 1) __hip_atomic_fetch_add((unsigned*)(sigp + NR * 128), 1u, __ATOMIC_RELAXED, __HIP_MEMORY_SCOPE_SYSTEM);
  }
  if (wid == 7) return;

  if (wid < 4) {
    const int wr = wid >> 1, wc = wid & 1;
    const unsigned call = sload(callp) + 1u;
    BAR();
    WAIT_LGKM(0);
    {
      char* ob = rel != 0 ? myws + WS_S + (size_t)((call & 1) * 256 + bid) * TILE1 + wid * (MF1 * 2048)
                          : P.out + ((size_t)(rank * MS + mh * M1 + wr * (M1 / 2)) * NS + ct * 128 + wc * 64) * 2;
      run_tile<MF1>(lds, wid, lane, ob, rel != 0 ? 128u : (unsigned)(NS * 2), tmo, call);
    }
    if (hasG) {
      const int blk = mt2 * 2 + wr;
      char* ob = P.out + ((size_t)((blk / 3) * MS + M1ROWS + (blk % 3) * 64) * NS + nl2 * 128 + wc * 64) * 2;
      run_tile<MF2>(lds, wid, lane, ob, NS * 2, tmo, call);
    }
    if (rel == 1) {
      waitv<0>();
      BAR();
    }
  } else if (wid < 4 + NDMA) {
    const int dw = wid - 4;
    unsigned va[PA1], vb[PB];
#pragma unroll
    for (int i = 0; i < PA1; ++i) {
      const int pc = i * NDMA + dw;
      va[i] = (unsigned)((pc * 8 + (lane >> 3)) * (KD * 2) + (((lane & 7) ^ ((4 * pc + (lane >> 4)) & 7)) * 16));
    }
#pragma unroll
    for (int i = 0; i < PB; ++i) {
      const int pc = i * NDMA + dw;
      vb[i] = (unsigned)((pc * 4 + (lane >> 4)) * (ND * 2) + (((lane & 15) ^ (((lane >> 4) << 2) | ((pc >> 1) & 1))) * 16));
    }
    const unsigned ldsb = (unsigned)(size_t)lds + dw * 1024;
    const char* a1 = (const char*)P.x + (size_t)(mh * M1) * KD * 2;
#define KX1 (mh | (((idx >> 1) & 3) << 1))
#define KX2 ((m6 ^ n4) | ((xcd & 3) << 3))
#define LROT 0
    const int kx1 = rel == 0 ? ((idx & 3) | ((xcd & 3) << 3)) : (KX1), kr1 = rel == 0 ? LROT : 0, kx2 = KX2;
#define LD_B(wb, kt, S) Pieces<0, PB, (S) * 16384, false>::go((wb) + (size_t)(kt) * 64 * ND * 2, vb, ldsb)
#define KT1(j) ((((j) + kr1) & (NKT - 1)) ^ kx1)
#define LD_1(j, S) do { const int kt_ = KT1(j); LD_B(w1, kt_, S); Pieces<0, PA1, A_OFF + (S) * ASL, false>::go(a1 + kt_ * 128, va, ldsb); } while (0)
#define LD_A2(j, S) do { Pieces<0, PA2 / 2, A_OFF + (S) * ASL, true>::go(a2lo + ((j) ^ kx2) * 128, va, ldsb); \
                         Pieces<PA2 / 2, PA2, A_OFF + (S) * ASL, true>::go(a2hi + ((j) ^ kx2) * 128, va, ldsb); } while (0)
#define QUAD2(j, W0, W1, W2, W3) do { \
      waitv<W0>(); BAR(); LD_B(w2, (j) ^ kx2, 0); LD_A2(j, 0); \
      waitv<W1>(); BAR(); LD_B(w2, ((j) + 1) ^ kx2, 1); LD_A2((j) + 1, 1); \
      waitv<W2>(); BAR(); LD_B(w2, ((j) + 2) ^ kx2, 2); LD_A2((j) + 2, 2); \
      waitv<W3>(); BAR(); LD_B(w2, ((j) + 3) ^ kx2, 3); LD_A2((j) + 3, 3); } while (0)
    constexpr int P1 = PA1 + PB, P2 = PA2 + PB;
    unsigned call;
    {
      const char* w1 = (const char*)P.w + (size_t)(dst1 * 16 + ct) * 256;
      LD_1(0, 0); LD_1(1, 1); LD_1(2, 2);
      call = sload(callp) + 1u;
      waitv<2 * P1>();
      if (rel == 0) {
        LD_1(3, 3);
        while (__builtin_amdgcn_s_memrealtime() - tk0 < 370) __builtin_amdgcn_s_sleep(2);
        const u64 c0 = __builtin_amdgcn_s_memtime(), r0 = __builtin_amdgcn_s_memrealtime();
        while (__builtin_amdgcn_s_memrealtime() - tk0 < 370 + CLKW * 52) __builtin_amdgcn_s_sleep(2);
        const u64 c1 = __builtin_amdgcn_s_memtime(), r1 = __builtin_amdgcn_s_memrealtime();
        const int pace = pace_of(c1 - c0, r1 - r0);
        while (__builtin_amdgcn_s_memrealtime() - tk0 < (u64)(NKT * pace + LOFF)) __builtin_amdgcn_s_sleep(4);
        BAR();
      } else {
        BAR();
        LD_1(3, 3);
      }
#pragma unroll 1
      for (int j = 4; j < NKT; j += 4) {
        waitv<2 * P1>(); BAR(); LD_1(j, 0);
        waitv<2 * P1>(); BAR(); LD_1(j + 1, 1);
        waitv<2 * P1>(); BAR(); LD_1(j + 2, 2);
        waitv<2 * P1>(); BAR(); LD_1(j + 3, 3);
      }
    }
    if (hasG) {
      const char* w2 = (const char*)P.w + (size_t)(rank * 16 + nl2) * 256;
      const int b0 = mt2 * 2, b1 = b0 + 1;
      const char* gb = myws + WS_G + (size_t)((call & 1) * NR) * XBLK;
      const char* xb = (const char*)P.x + (size_t)M1ROWS * KD * 2;
      const char* a2lo = (b0 / 3 == rank) ? xb + (size_t)((b0 % 3) * 64) * KD * 2 : gb + (size_t)(b0 * 64) * KD * 2;
      const char* a2hi = ((b1 / 3 == rank) ? xb + (size_t)((b1 % 3) * 64) * KD * 2 : gb + (size_t)(b1 * 64) * KD * 2) - (size_t)64 * KD * 2;
      QUAD2(0, 2 * P1, P1 + P2, 2 * P2, 2 * P2);
#pragma unroll 1
      for (int j = 4; j < NKT; j += 4) QUAD2(j, 2 * P2, 2 * P2, 2 * P2, 2 * P2);
      waitv<2 * P2>(); BAR();
      waitv<P2>(); BAR();
    } else {
      waitv<2 * P1>(); BAR();
      waitv<P1>(); BAR();
    }
    waitv<0>(); BAR();
    BAR();
    if (rel == 1) BAR();
  } else {
    int pcount = rel == 0 ? 1000 : 0, pace = PACE_COOL;
    const unsigned call = sload(callp) + 1u;
    unsigned* rdy = (unsigned*)(P.ws[dst1] + WS_OF + (size_t)(rank * 32 + t32) * 128);
    const int s0 = (mt2 * 2) / 3, s1 = (mt2 * 2 + 1) / 3;
    auto gate = [&](int h, int nb) {
      const unsigned* c0 = (const unsigned*)(myws + WS_XC + (h * NR + s0) * 128);
      const unsigned* c1 = (const unsigned*)(myws + WS_XC + (h * NR + s1) * 128);
      const unsigned v0 = __hip_atomic_load(c0, __ATOMIC_RELAXED, __HIP_MEMORY_SCOPE_SYSTEM);
      const unsigned v1 = __hip_atomic_load(c1, __ATOMIC_RELAXED, __HIP_MEMORY_SCOPE_SYSTEM);
      SCHED();
#pragma unroll 1
      for (int i = 0; i < nb; ++i) BAR6();
      SCHED();
      if (s0 != rank && rfl(v0) < call * 192u) wait_ge(c0, call * 192u, tmo, call);
      if (s1 != rank && rfl(v1) < call * 192u) wait_ge(c1, call * 192u, tmo, call);
    };
    auto bars = [&](int n) {
#pragma unroll 1
      for (int i = 0; i < n; ++i) BAR6();
    };
    if (rel != 0) {
      bars(3);
      const u64 c0 = __builtin_amdgcn_s_memtime(), r0 = __builtin_amdgcn_s_memrealtime();
      bars(CLKW);
      const u64 c1 = __builtin_amdgcn_s_memtime(), r1 = __builtin_amdgcn_s_memrealtime();
      pace = pace_of(c1 - c0, r1 - r0);
    }
    if (rel == 0) {
      bars(NKT + 1);
    } else if (rel == 1) {
      bars(NKT + 2 - 3 - CLKW);
      if (lane == 0) __hip_atomic_store(rdy, call, __ATOMIC_RELAXED, __HIP_MEMORY_SCOPE_SYSTEM);
    } else {
      bars(NKT - 3 - 4 - 3 - CLKW);
      gate(0, 4);
      bars(5 + FLAGK);
      if (lane == 0) __hip_atomic_store(rdy, call, __ATOMIC_RELAXED, __HIP_MEMORY_SCOPE_SYSTEM);
      bars(23 - FLAGK);
      gate(1, 4);
      bars(36);
    }
  }

  if (rel == 1) {
    const unsigned call = sload(callp) + 1u;
    const int src = (rank + 1 + wid) & (NR - 1);
    bool ok = true;
    ok = wait_ge((const unsigned*)(myws + WS_OF + (size_t)(src * 32 + t32) * 128), call, tmo, call);
    const char* sp = P.ws[src] + WS_S + (size_t)((call & 1) * 256 + rank * 32 + t32) * TILE1;
    char* ob = P.out + ((size_t)(src * MS + mh * M1) * NS + rcol(rank) * 128) * 2;
    const unsigned vl = lane * 16, vrow = (unsigned)((lane >> 3) * (NS * 2) + (lane & 7) * 16);
    if (ok) {
      pull_tile<PDEP>(sp, ob, vl, vrow);
    } else {
#pragma unroll 1
      for (int pc = 0; pc < OPC; ++pc) {
        const int w = pc / (2 * MF1), r = pc % (2 * MF1);
        store_sys(ob, vrow + (((w >> 1) * (M1 / 2) + r * 8) * NS + (w & 1) * 64) * 2,
                  (u32x4){0x7fc07fc0u, 0x7fc07fc0u, 0x7fc07fc0u, 0x7fc07fc0u});
      }
    }
    BAR();
  }
  if (wid == 6 && lane == 0) {
    __hip_atomic_fetch_add((unsigned*)(myws + WS_CALL) + bid, 1u, __ATOMIC_RELAXED, __HIP_MEMORY_SCOPE_AGENT);
  }
}

}

extern "C" void kernel_launch(int rank, int nranks, ncclComm_t comm,
                              hipStream_t stream,
                              void* const* d_in, const int* in_sizes, int n_in,
                              void* d_out, int out_size,
                              const int* in_dtypes, int out_dtype,
                              void* const* d_out_all,
                              void* const* d_ws_all, size_t ws_bytes) {
  if (nranks != NR || (unsigned)rank >= (unsigned)NR || ws_bytes < WS_NEED || n_in < 2) {
    fprintf(stderr, "unsupported configuration\n");
    return;
  }
  static bool init[NR] = {};
  if (!init[rank]) {
    HIP_CHECK(hipFuncSetAttribute((const void*)fused_gemm_a2a, hipFuncAttributeMaxDynamicSharedMemorySize, LDS_BYTES));
    init[rank] = true;
  }
  Params P;
  P.x = (const u16*)d_in[0];
  P.w = (const u16*)d_in[1];
  P.out = (char*)d_out;
  for (int p = 0; p < NR; ++p) P.ws[p] = (char*)d_ws_all[p];
  P.rank = rank;
  hipLaunchKernelGGL(fused_gemm_a2a, dim3(256), dim3(NTHREADS), LDS_BYTES, stream, P);
  HIP_CHECK(hipGetLastError());
}
```

```cpp
#include <hip/hip_runtime.h>
#include <rccl/rccl.h>
#include <cstdio>

#define HIP_CHECK(expr) do { hipError_t _e = (expr); if (_e != hipSuccess) { \
    fprintf(stderr, "HIP error %s at %s:%d\n", hipGetErrorString(_e), __FILE__, __LINE__); \
    return; } } while (0)

namespace {

typedef unsigned short u16;
typedef unsigned long long u64;
typedef unsigned u32x4 __attribute__((ext_vector_type(4)));
typedef short s16x4 __attribute__((ext_vector_type(4)));
typedef short s16x8 __attribute__((ext_vector_type(8)));
typedef __bf16 bf16x8 __attribute__((ext_vector_type(8)));
typedef float f32x4 __attribute__((ext_vector_type(4)));
#define LAS __attribute__((address_space(3)))

constexpr int NR = 8;
constexpr int MS = 512, KD = 4096, ND = 16384, NS = 2048;
constexpr int MF1 = 5, MF2 = 4;
constexpr int M1 = 32 * MF1, M2 = 32 * MF2;
constexpr int M1ROWS = 2 * M1, XROWS = MS - M1ROWS;
constexpr int NKT = KD / 64;
constexpr int NDMA = 2;
constexpr int NWAVES = 8;
constexpr int NTHREADS = NWAVES * 64;
constexpr int PA1 = M1 / 8 / NDMA, PA2 = M2 / 8 / NDMA, PB = 16 / NDMA;
constexpr int A_OFF = 65536, ASL = M1 * 128, E_OFF = A_OFF + 4 * ASL;
constexpr int LDS_BYTES = E_OFF + 16384;
static_assert(LDS_BYTES <= 163840, "lds");
constexpr int XBLK = XROWS * KD * 2;
constexpr int OPC = M1 / 4;
constexpr int TILE1 = OPC * 1024;
static_assert(XROWS == 6 * 32 && NR * XROWS == 12 * M2, "groups");

constexpr size_t WS_CALL = 0;
constexpr size_t WS_TMO = 4096;
constexpr size_t WS_XC = 8192;
constexpr size_t WS_OF = 16384;
constexpr size_t WS_G = 1u << 20;
constexpr size_t WS_S = WS_G + (size_t)2 * NR * XBLK;
constexpr size_t WS_NEED = WS_S + (size_t)2 * 256 * TILE1;

constexpr u64 DEADLINE = 100000000ull;
#define XTHR 2
#define PDEP 1
#define FLAGK 0

#define PACE_COOL 54
#define PACE_HOT 58
#define CLK_COOL 170
#define CLK_HOT 178
#define CLKW 12
#define LOFF 90
#define PACE0 0

struct Params {
  const u16* x; const u16* w;
  char* out; char* ws[NR];
  int rank;
};

#define BAR() do { asm volatile("" ::: "memory"); __builtin_amdgcn_s_barrier(); asm volatile("" ::: "memory"); } while (0)
#define PACE() do { if (++pcount <= NKT + 1) { while (__builtin_amdgcn_s_memrealtime() - tk0 < (u64)(PACE0 + pstag + pcount * pace)) __builtin_amdgcn_s_sleep(1); } } while (0)
#define BAR6() do { PACE(); BAR(); } while (0)
#define SCHED() __builtin_amdgcn_sched_barrier(0)
#define WAIT_LGKM(n) __builtin_amdgcn_s_waitcnt(0xC07F | ((n) << 8))
#define MFMA_DRAIN() asm volatile("s_nop 7\n\ts_nop 7\n\ts_nop 3" ::: "memory")
template <int N> __device__ __forceinline__ void waitv() { static_assert(N <= 63, "vmcnt"); asm volatile("s_waitcnt vmcnt(%0)" :: "i"(N) : "memory"); }

template <int OFF, bool SYS> __device__ __forceinline__ void glds(const char* sbase, unsigned voff, unsigned ldsbase) {
  if (SYS) asm volatile("s_add_u32 m0, %2, %3\n\ts_nop 0\n\tglobal_load_lds_dwordx4 %0, %1 sc0 sc1" :: "v"(voff), "s"(sbase), "s"(ldsbase), "i"(OFF) : "memory", "scc");
  else asm volatile("s_add_u32 m0, %2, %3\n\ts_nop 0\n\tglobal_load_lds_dwordx4 %0, %1" :: "v"(voff), "s"(sbase), "s"(ldsbase), "i"(OFF) : "memory", "scc");
}
template <int I, int N, int BASE, bool SYS> struct Pieces {
  static __device__ __forceinline__ void go(const char* sbase, const unsigned* voff, unsigned ldsbase) {
    glds<BASE + I * 1024 * NDMA, SYS>(sbase, voff[I], ldsbase);
    Pieces<I + 1, N, BASE, SYS>::go(sbase, voff, ldsbase);
  }
};
template <int N, int BASE, bool SYS> struct Pieces<N, N, BASE, SYS> { static __device__ __forceinline__ void go(const char*, const unsigned*, unsigned) {} };

__device__ __forceinline__ void store_sys(char* sbase, unsigned voff, u32x4 v) {
  asm volatile("global_store_dwordx4 %0, %1, %2 sc0 sc1\n\ts_nop 1" :: "v"(voff), "v"(v), "s"(sbase) : "memory");
}
__device__ __forceinline__ unsigned cvt_pk_bf16(float lo, float hi) {
  unsigned r; asm("v_cvt_pk_bf16_f32 %0, %1, %2" : "=v"(r) : "v"(lo), "v"(hi)); return r;
}
__device__ __forceinline__ float relu_nan(float v) { return __builtin_elementwise_maximum(v, 0.f); }
__device__ __forceinline__ unsigned rfl(unsigned v) { return (unsigned)__builtin_amdgcn_readfirstlane(v); }
__device__ __forceinline__ unsigned sload(const char* p) {
  unsigned r; asm volatile("s_load_dword %0, %1, 0x0\n\ts_waitcnt lgkmcnt(0)" : "=s"(r) : "s"(p) : "memory"); return r;
}

__device__ __forceinline__ int pace_of(u64 cycles, u64 ticks) {
  const int f = (int)((unsigned)cycles * 10u / ((unsigned)ticks | 1u));
  const int p = PACE_COOL + (f - CLK_COOL) * (PACE_HOT - PACE_COOL) / (CLK_HOT - CLK_COOL);
  return p < PACE_COOL ? PACE_COOL : p > PACE_HOT ? PACE_HOT : p;
}

__device__ __forceinline__ bool wait_ge(const unsigned* flag, unsigned want, unsigned* tmo, unsigned call) {
  const u64 t0 = __builtin_amdgcn_s_memrealtime();
  unsigned polls = 0;
  while (rfl(__hip_atomic_load(flag, __ATOMIC_RELAXED, __HIP_MEMORY_SCOPE_SYSTEM)) < want) {
    if ((++polls & 15u) == 0u &&
        (rfl(__hip_atomic_load(tmo, __ATOMIC_RELAXED, __HIP_MEMORY_SCOPE_AGENT)) == call ||
         __builtin_amdgcn_s_memrealtime() - t0 > DEADLINE)) {
      __hip_atomic_store(tmo, call, __ATOMIC_RELAXED, __HIP_MEMORY_SCOPE_AGENT);
      waitv<0>();
      return false;
    }
    __builtin_amdgcn_s_sleep(2);
  }
  return true;
}

template <int MF> struct Frag { bf16x8 a[MF]; s16x4 lo[4], hi[4]; };

template <int MF>
__device__ __forceinline__ void read_one(Frag<MF>& f, int i, LAS char* lds, int s, int h, const unsigned (&ra)[2], const unsigned (&rb)[4]) {
  if (i >= 2 && i < 2 + MF) {
    f.a[i - 2] = *(LAS bf16x8*)(lds + ra[h] + s * ASL + (i - 2) * 2048);
  } else {
    const int k = i < 2 ? i : i - MF, bn = k >> 1;
    s16x4 v = __builtin_amdgcn_ds_read_tr16_b64_v4i16((LAS s16x4*)(lds + rb[bn] + s * 16384 + h * 8192 + (k & 1) * 1024));
    if (k & 1) f.hi[bn] = v; else f.lo[bn] = v;
  }
}
template <int MF, bool RD>
__device__ __forceinline__ void half_step(f32x4 (&acc)[MF][4], const Frag<MF>& f, Frag<MF>& g, LAS char* lds, int s, int h,
                                          const unsigned (&ra)[2], const unsigned (&rb)[4]) {
#pragma unroll
  for (int i = 0; i < 4 * MF; ++i) {
    const int bn = i / MF, mi = (bn & 1) ? MF - 1 - i % MF : i % MF;
    const s16x8 bv = __builtin_shufflevector(f.lo[bn], f.hi[bn], 0, 1, 2, 3, 4, 5, 6, 7);
    asm volatile("v_mfma_f32_16x16x32_bf16 %0, %1, %2, %0" : "+v"(acc[mi][bn]) : "v"(bv), "v"(f.a[mi]));
    if (RD && i < 8 + MF) read_one<MF>(g, i, lds, s, h, ra, rb);
    SCHED();
  }
}
template <int MF, int S, bool NEXT>
__device__ __forceinline__ void ktile(f32x4 (&acc)[MF][4], Frag<MF>& f0, Frag<MF>& f1, LAS char* lds,
                                      const unsigned (&ra)[2], const unsigned (&rb)[4]) {
  WAIT_LGKM(0);
  half_step<MF, true>(acc, f0, f1, lds, S, 1, ra, rb);
  WAIT_LGKM(0);
  if (S == FLAGK) waitv<0>();
  BAR();
  SCHED();
  half_step<MF, NEXT>(acc, f1, f0, lds, (S + 1) & 3, 0, ra, rb);
}
__device__ __forceinline__ u32x4 pack8(const f32x4& v0, const f32x4& v1, bool odd) {
  unsigned a0 = cvt_pk_bf16(relu_nan(v0[0]), relu_nan(v0[1])), a1 = cvt_pk_bf16(relu_nan(v0[2]), relu_nan(v0[3]));
  unsigned b0 = cvt_pk_bf16(relu_nan(v1[0]), relu_nan(v1[1])), b1 = cvt_pk_bf16(relu_nan(v1[2]), relu_nan(v1[3]));
  u32x4 o;
  o[0] = odd ? b0 : a0; o[1] = odd ? b1 : a1; o[2] = odd ? a0 : b0; o[3] = odd ? a1 : b1;
  return o;
}
template <int MF>
__device__ __forceinline__ void run_tile(LAS char* lds, int wid, int lane, char* ob, unsigned pitch, const unsigned* tmo, unsigned call) {
  const int wr = wid >> 1, wc = wid & 1, fr = lane & 15, fq = lane >> 4, q = fr >> 2, p = lane & 3;
  const unsigned rb00 = (8 * fq + q) * 256 + ((((2 * wc) ^ q) << 2) | (p ^ (fq & 1))) * 16 + (p & 1) * 8;
  const unsigned rb[4] = {rb00, rb00 ^ 8u, rb00 ^ 64u, rb00 ^ 72u};
  const unsigned rab = A_OFF + (wr * (MF * 16) + fr) * 128 + (unsigned)((fq ^ (fr >> 1)) * 16);
  const unsigned ra[2] = {rab, rab ^ 64u};
  Frag<MF> f0, f1;
#pragma unroll
  for (int i = 0; i < 8 + MF; ++i) read_one<MF>(f0, i, lds, 0, 0, ra, rb);
  f32x4 acc[MF][4];
#pragma unroll
  for (int i = 0; i < MF; ++i)
#pragma unroll
    for (int b = 0; b < 4; ++b) { acc[i][b] = (f32x4){0.f, 0.f, 0.f, 0.f}; asm volatile("" : "+v"(acc[i][b])); }
  asm volatile("s_nop 1" ::: "memory");
  SCHED();
  for (int j = 0; j < NKT - 4; j += 4) {
    ktile<MF, 0, true>(acc, f0, f1, lds, ra, rb);
    ktile<MF, 1, true>(acc, f0, f1, lds, ra, rb);
    ktile<MF, 2, true>(acc, f0, f1, lds, ra, rb);
    ktile<MF, 3, true>(acc, f0, f1, lds, ra, rb);
  }
  static_assert(FLAGK == 0, "the load below stays in flight until the epilogue");
  ktile<MF, 0, true>(acc, f0, f1, lds, ra, rb);
  unsigned tmo_v = call + 1u;
  if (MF == MF2) tmo_v = __hip_atomic_load(tmo, __ATOMIC_RELAXED, __HIP_MEMORY_SCOPE_AGENT);
  SCHED();
  ktile<MF, 1, true>(acc, f0, f1, lds, ra, rb);
  ktile<MF, 2, true>(acc, f0, f1, lds, ra, rb);
  ktile<MF, 3, false>(acc, f0, f1, lds, ra, rb);
  MFMA_DRAIN();
  SCHED();
  const bool odd = fq & 1, bad = tmo_v == call;
  LAS char* sc = lds + E_OFF + wid * 4096;
  const unsigned wo = fr * 128 + ((fq ^ (fr & 7)) * 16), ro = (lane >> 3) * 128 + (((lane & 7) ^ (lane >> 3)) * 16);
  unsigned vo = (lane >> 3) * pitch + (lane & 7) * 16;
#pragma unroll
  for (int m0 = 0; m0 < MF; m0 += 2) {
#pragma unroll
    for (int mi = m0; mi < m0 + 2 && mi < MF; ++mi) {
      *(LAS u32x4*)(sc + (mi - m0) * 2048 + wo) = pack8(acc[mi][0], acc[mi][1], odd);
      *(LAS u32x4*)(sc + (mi - m0) * 2048 + (wo ^ 64u)) = pack8(acc[mi][2], acc[mi][3], odd);
    }
    asm volatile("" ::: "memory");
#pragma unroll
    for (int r = 0; r < (m0 + 2 <= MF ? 4 : 2); ++r) {
      u32x4 o = *(LAS u32x4*)(sc + r * 1024 + ro);
      if (MF == MF2 && bad) o = (u32x4){0x7fc07fc0u, 0x7fc07fc0u, 0x7fc07fc0u, 0x7fc07fc0u};
      store_sys(ob, vo, o);
      vo += 8 * pitch;
    }
    asm volatile("" ::: "memory");
  }
}

template <int D>
__device__ __forceinline__ void pull_tile(const char* sp, char* ob, unsigned vl, unsigned vrow) {
  u32x4 v[D + 1];
#pragma unroll
  for (int i = 0; i < OPC + D; ++i) {
    if (i < OPC) asm volatile("global_load_dwordx4 %0, %1, %2 sc0 sc1" : "=&v"(v[i % (D + 1)]) : "v"(vl + i * 1024), "s"(sp) : "memory");
    if (i >= D) {
      const int pc = i - D, w = pc / (2 * MF1), r = pc % (2 * MF1);
      const int nld = (i < OPC ? i : OPC - 1) - pc, nst = pc < D ? pc : D;
      asm volatile("s_waitcnt vmcnt(%1)" : "+v"(v[pc % (D + 1)]) : "i"(nld + nst) : "memory");
      store_sys(ob, vrow + (((w >> 1) * (M1 / 2) + r * 8) * NS + (w & 1) * 64) * 2, v[pc % (D + 1)]);
    }
  }
}


__global__ void __launch_bounds__(NTHREADS, 2) fused_gemm_a2a(const Params P) {
  extern __shared__ __attribute__((aligned(16))) char smem_[];
  LAS char* lds = (LAS char*)smem_;
  const int tid = threadIdx.x, lane = tid & 63, wid = __builtin_amdgcn_readfirstlane(tid >> 6);
  const int bid = blockIdx.x, xcd = bid & 7, idx = bid >> 3, rank = P.rank;
  const int dst1 = idx >> 2, t32 = bid & 31;
  const int rel = (dst1 - rank) & (NR - 1);
#define CMIX 1
  auto rcol = [&](int owner) { const int hb = (t32 >> 4) & 1; return hb * 8 + ((xcd + CMIX * (owner * 2 + hb)) & 7); };
  const int mh = rel == 0 ? (xcd & 1) : (idx & 1), ct = rel == 0 ? (xcd >> 1) * 4 + (idx & 3) : rcol(dst1);
  const bool hasG = rel >= 2;
  const int ci = (rel < 2 ? 0 : rel - 2) * 4 + (idx & 3), m6 = ci % 6, n4 = ci / 6;
  const int mt2 = (xcd & 1) * 6 + m6, nl2 = (xcd >> 1) * 4 + n4;
  char* myws = P.ws[rank];
  unsigned* tmo = (unsigned*)(myws + WS_TMO);
  const char* callp = myws + WS_CALL + bid * 4;
  const u64 tk0 = __builtin_amdgcn_s_memrealtime();

  if (rel == 0 && (wid < 4 || wid >= 6)) {
    static_assert(XTHR < NR - 1, "xthr");
    const int xw = wid < 4 ? wid : wid - 2, row = t32 * 6 + xw;
    const unsigned vl = lane * 16;
    u32x4 xv[8];
    const char* xs = (const char*)P.x + (size_t)(M1ROWS + row) * KD * 2 + vl;
#pragma unroll
    for (int i = 0; i < 8; ++i) xv[i] = *(const u32x4*)(xs + i * 1024);
    const unsigned call = sload(callp) + 1u;
    const size_t goff = WS_G + (size_t)((call & 1) * NR + rank) * XBLK + (size_t)row * KD * 2;
    char* g[NR - 1];
#pragma unroll
    for (int j = 0; j < NR - 1; ++j) { int pj = j + xw; if (pj >= NR - 1) pj -= NR - 1; g[j] = P.ws[(rank + 1 + pj) & (NR - 1)] + goff; }
    char* sigp;
    {
      const int peer = (rank + 1 + lane) & (NR - 1);
      char* pw = P.ws[0];
#pragma unroll
      for (int r = 1; r < NR; ++r) pw = (peer == r) ? P.ws[r] : pw;
      sigp = pw + WS_XC + rank * 128;
    }
#pragma unroll
    for (int i = 0; i < 8; ++i) asm volatile("" : "+v"(xv[i]));
#pragma unroll
    for (int i = 0; i < 8; ++i) {
      unsigned vo = vl + i * 1024;
      asm volatile("" : "+v"(vo));
#pragma unroll
      for (int j = 0; j < NR - 1; ++j) { store_sys(g[j], vo, xv[i]); waitv<XTHR>(); }
      if (i == 4 && lane < NR - 1) __hip_atomic_fetch_add((unsigned*)sigp, 1u, __ATOMIC_RELAXED, __HIP_MEMORY_SCOPE_SYSTEM);
    }
    waitv<0>();
    if (lane < NR - 1) __hip_atomic_fetch_add((unsigned*)(sigp + NR * 128), 1u, __ATOMIC_RELAXED, __HIP_MEMORY_SCOPE_SYSTEM);
  }
  if (wid == 7) return;

  if (wid < 4) {
    const int wr = wid >> 1, wc = wid & 1;
    const unsigned call = sload(callp) + 1u;
    BAR();
    WAIT_LGKM(0);
    {
      char* ob = rel != 0 ? myws + WS_S + (size_t)((call & 1) * 256 + bid) * TILE1 + wid * (MF1 * 2048)
                          : P.out + ((size_t)(rank * MS + mh * M1 + wr * (M1 / 2)) * NS + ct * 128 + wc * 64) * 2;
      run_tile<MF1>(lds, wid, lane, ob, rel != 0 ? 128u : (unsigned)(NS * 2), tmo, call);
    }
    if (hasG) {
      const int blk = mt2 * 2 + wr;
      char* ob = P.out + ((size_t)((blk / 3) * MS + M1ROWS + (blk % 3) * 64) * NS + nl2 * 128 + wc * 64) * 2;
      run_tile<MF2>(lds, wid, lane, ob, NS * 2, tmo, call);
    }
    if (rel == 1) {
      waitv<0>();
      BAR();
    }
  } else if (wid < 4 + NDMA) {
    const int dw = wid - 4;
    unsigned va[PA1], vb[PB];
#pragma unroll
    for (int i = 0; i < PA1; ++i) {
      const int pc = i * NDMA + dw;
      va[i] = (unsigned)((pc * 8 + (lane >> 3)) * (KD * 2) + (((lane & 7) ^ ((4 * pc + (lane >> 4)) & 7)) * 16));
    }
#pragma unroll
    for (int i = 0; i < PB; ++i) {
      const int pc = i * NDMA + dw;
      vb[i] = (unsigned)((pc * 4 + (lane >> 4)) * (ND * 2) + (((lane & 15) ^ (((lane >> 4) << 2) | ((pc >> 1) & 1))) * 16));
    }
    const unsigned ldsb = (unsigned)(size_t)lds + dw * 1024;
    const char* a1 = (const char*)P.x + (size_t)(mh * M1) * KD * 2;
#define KX1 (mh | (((idx >> 1) & 3) << 1))
#define KX2 ((m6 ^ n4) | ((xcd & 3) << 3))
#define LROT 0
    const int kx1 = rel == 0 ? ((idx & 3) | ((xcd & 3) << 3)) : (KX1), kr1 = rel == 0 ? LROT : 0, kx2 = KX2;
#define LD_B(wb, kt, S) Pieces<0, PB, (S) * 16384, false>::go((wb) + (size_t)(kt) * 64 * ND * 2, vb, ldsb)
#define KT1(j) ((((j) + kr1) & (NKT - 1)) ^ kx1)
#define LD_1(j, S) do { const int kt_ = KT1(j); LD_B(w1, kt_, S); Pieces<0, PA1, A_OFF + (S) * ASL, false>::go(a1 + kt_ * 128, va, ldsb); } while (0)
#define LD_A2(j, S) do { Pieces<0, PA2 / 2, A_OFF + (S) * ASL, true>::go(a2lo + ((j) ^ kx2) * 128, va, ldsb); \
                         Pieces<PA2 / 2, PA2, A_OFF + (S) * ASL, true>::go(a2hi + ((j) ^ kx2) * 128, va, ldsb); } while (0)
#define QUAD2(j, W0, W1, W2, W3) do { \
      waitv<W0>(); BAR(); LD_B(w2, (j) ^ kx2, 0); LD_A2(j, 0); \
      waitv<W1>(); BAR(); LD_B(w2, ((j) + 1) ^ kx2, 1); LD_A2((j) + 1, 1); \
      waitv<W2>(); BAR(); LD_B(w2, ((j) + 2) ^ kx2, 2); LD_A2((j) + 2, 2); \
      waitv<W3>(); BAR(); LD_B(w2, ((j) + 3) ^ kx2, 3); LD_A2((j) + 3, 3); } while (0)
    constexpr int P1 = PA1 + PB, P2 = PA2 + PB;
    unsigned call;
    {
      const char* w1 = (const char*)P.w + (size_t)(dst1 * 16 + ct) * 256;
      LD_1(0, 0); LD_1(1, 1); LD_1(2, 2);
      call = sload(callp) + 1u;
      waitv<2 * P1>();
      if (rel == 0) {
        LD_1(3, 3);
        while (__builtin_amdgcn_s_memrealtime() - tk0 < 370) __builtin_amdgcn_s_sleep(2);
        const u64 c0 = __builtin_amdgcn_s_memtime(), r0 = __builtin_amdgcn_s_memrealtime();
        while (__builtin_amdgcn_s_memrealtime() - tk0 < 370 + CLKW * 52) __builtin_amdgcn_s_sleep(2);
        const u64 c1 = __builtin_amdgcn_s_memtime(), r1 = __builtin_amdgcn_s_memrealtime();
        const int pace = pace_of(c1 - c0, r1 - r0);
        while (__builtin_amdgcn_s_memrealtime() - tk0 < (u64)(NKT * pace + LOFF)) __builtin_amdgcn_s_sleep(4);
        BAR();
      } else {
        BAR();
        LD_1(3, 3);
      }
#pragma unroll 1
      for (int j = 4; j < NKT; j += 4) {
        waitv<2 * P1>(); BAR(); LD_1(j, 0);
        waitv<2 * P1>(); BAR(); LD_1(j + 1, 1);
        waitv<2 * P1>(); BAR(); LD_1(j + 2, 2);
        waitv<2 * P1>(); BAR(); LD_1(j + 3, 3);
      }
    }
    if (hasG) {
      const char* w2 = (const char*)P.w + (size_t)(rank * 16 + nl2) * 256;
      const int b0 = mt2 * 2, b1 = b0 + 1;
      const char* gb = myws + WS_G + (size_t)((call & 1) * NR) * XBLK;
      const char* xb = (const char*)P.x + (size_t)M1ROWS * KD * 2;
      const char* a2lo = (b0 / 3 == rank) ? xb + (size_t)((b0 % 3) * 64) * KD * 2 : gb + (size_t)(b0 * 64) * KD * 2;
      const char* a2hi = ((b1 / 3 == rank) ? xb + (size_t)((b1 % 3) * 64) * KD * 2 : gb + (size_t)(b1 * 64) * KD * 2) - (size_t)64 * KD * 2;
      QUAD2(0, 2 * P1, P1 + P2, 2 * P2, 2 * P2);
#pragma unroll 1
      for (int j = 4; j < NKT; j += 4) QUAD2(j, 2 * P2, 2 * P2, 2 * P2, 2 * P2);
      waitv<2 * P2>(); BAR();
      waitv<P2>(); BAR();
    } else {
      waitv<2 * P1>(); BAR();
      waitv<P1>(); BAR();
    }
    waitv<0>(); BAR();
    BAR();
    if (rel == 1) BAR();
  } else {
    int pcount = rel == 0 ? 1000 : 0, pace = PACE_COOL;
#define PSTAGV 108
    const int pstag = ((((idx >> 1) * 5 + xcd * 2) & 15) * PSTAGV) >> 4;
    const unsigned call = sload(callp) + 1u;
    unsigned* rdy = (unsigned*)(P.ws[dst1] + WS_OF + (size_t)(rank * 32 + t32) * 128);
    const int s0 = (mt2 * 2) / 3, s1 = (mt2 * 2 + 1) / 3;
    auto gate = [&](int h, int nb) {
      const unsigned* c0 = (const unsigned*)(myws + WS_XC + (h * NR + s0) * 128);
      const unsigned* c1 = (const unsigned*)(myws + WS_XC + (h * NR + s1) * 128);
      const unsigned v0 = __hip_atomic_load(c0, __ATOMIC_RELAXED, __HIP_MEMORY_SCOPE_SYSTEM);
      const unsigned v1 = __hip_atomic_load(c1, __ATOMIC_RELAXED, __HIP_MEMORY_SCOPE_SYSTEM);
      SCHED();
#pragma unroll 1
      for (int i = 0; i < nb; ++i) BAR6();
      SCHED();
      if (s0 != rank && rfl(v0) < call * 192u) wait_ge(c0, call * 192u, tmo, call);
      if (s1 != rank && rfl(v1) < call * 192u) wait_ge(c1, call * 192u, tmo, call);
    };
    auto bars = [&](int n) {
#pragma unroll 1
      for (int i = 0; i < n; ++i) BAR6();
    };
    if (rel != 0) {
      bars(3);
      const u64 c0 = __builtin_amdgcn_s_memtime(), r0 = __builtin_amdgcn_s_memrealtime();
      bars(CLKW);
      const u64 c1 = __builtin_amdgcn_s_memtime(), r1 = __builtin_amdgcn_s_memrealtime();
      pace = pace_of(c1 - c0, r1 - r0);
    }
    if (rel == 0) {
      bars(NKT + 1);
    } else if (rel == 1) {
      bars(NKT + 2 - 3 - CLKW);
      if (lane == 0) __hip_atomic_store(rdy, call, __ATOMIC_RELAXED, __HIP_MEMORY_SCOPE_SYSTEM);
    } else {
      bars(NKT - 3 - 4 - 3 - CLKW);
      gate(0, 4);
      bars(5 + FLAGK);
      if (lane == 0) __hip_atomic_store(rdy, call, __ATOMIC_RELAXED, __HIP_MEMORY_SCOPE_SYSTEM);
      bars(23 - FLAGK);
      gate(1, 4);
      bars(36);
    }
  }

  if (rel == 1) {
    const unsigned call = sload(callp) + 1u;
    const int src = (rank + 1 + wid) & (NR - 1);
    bool ok = true;
    ok = wait_ge((const unsigned*)(myws + WS_OF + (size_t)(src * 32 + t32) * 128), call, tmo, call);
    const char* sp = P.ws[src] + WS_S + (size_t)((call & 1) * 256 + rank * 32 + t32) * TILE1;
    char* ob = P.out + ((size_t)(src * MS + mh * M1) * NS + rcol(rank) * 128) * 2;
    const unsigned vl = lane * 16, vrow = (unsigned)((lane >> 3) * (NS * 2) + (lane & 7) * 16);
    if (ok) {
      pull_tile<PDEP>(sp, ob, vl, vrow);
    } else {
#pragma unroll 1
      for (int pc = 0; pc < OPC; ++pc) {
        const int w = pc / (2 * MF1), r = pc % (2 * MF1);
        store_sys(ob, vrow + (((w >> 1) * (M1 / 2) + r * 8) * NS + (w & 1) * 64) * 2,
                  (u32x4){0x7fc07fc0u, 0x7fc07fc0u, 0x7fc07fc0u, 0x7fc07fc0u});
      }
    }
    BAR();
  }
  if (wid == 6 && lane == 0) {
    __hip_atomic_fetch_add((unsigned*)(myws + WS_CALL) + bid, 1u, __ATOMIC_RELAXED, __HIP_MEMORY_SCOPE_AGENT);
  }
}

}

extern "C" void kernel_launch(int rank, int nranks, ncclComm_t comm,
                              hipStream_t stream,
                              void* const* d_in, const int* in_sizes, int n_in,
                              void* d_out, int out_size,
                              const int* in_dtypes, int out_dtype,
                              void* const* d_out_all,
                              void* const* d_ws_all, size_t ws_bytes) {
  if (nranks != NR || (unsigned)rank >= (unsigned)NR || ws_bytes < WS_NEED || n_in < 2) {
    fprintf(stderr, "unsupported configuration\n");
    return;
  }
  static bool init[NR] = {};
  if (!init[rank]) {
    HIP_CHECK(hipFuncSetAttribute((const void*)fused_gemm_a2a, hipFuncAttributeMaxDynamicSharedMemorySize, LDS_BYTES));
    init[rank] = true;
  }
  Params P;
  P.x = (const u16*)d_in[0];
  P.w = (const u16*)d_in[1];
  P.out = (char*)d_out;
  for (int p = 0; p < NR; ++p) P.ws[p] = (char*)d_ws_all[p];
  P.rank = rank;
  hipLaunchKernelGGL(fused_gemm_a2a, dim3(256), dim3(NTHREADS), LDS_BYTES, stream, P);
  HIP_CHECK(hipGetLastError());
}
```
